```python
import numpy as np

kernel_name = "gpu_ag_gemm_m4096_k4096_n4096_bf16_reluq8"


ABSMAX_RTOL = 0.02

COMPUTE_DTYPE = "bfloat16"
EPILOGUE = "relu_quant_int8"
IN_QUANT = None
OUT_QUANT = "int8"
IN_DTYPES = {"x": "bfloat16", "w_mat": "bfloat16"}
WORLD_SIZE = 8

def _to_bf16(x):
    u = np.ascontiguousarray(x, dtype=np.float32).view(np.uint32)
    return ((u + 0x7FFF + ((u >> 16) & 1)) & 0xFFFF0000).view(np.float32)

def setup_inputs():

    x = _to_bf16(np.random.randn(4096, 4096).astype(np.float32))
    w_mat = _to_bf16((np.random.randn(4096, 4096) / np.sqrt(4096)).astype(np.float32))
    return {"x": x, "w_mat": w_mat}

def _quant_dequant(y, amax):
    scale = amax / 127.0
    q = np.clip(np.round(y / scale), -127.0, 127.0)
    return (q * scale).astype(np.float32)
def _epilogue(y):
    y = np.maximum(y, 0.0)
    return _quant_dequant(y, np.abs(y).max())

def reference(x, w_mat):
    y = (x @ w_mat).astype(np.float32)
    return _epilogue(y)
```

```cpp
#include <hip/hip_runtime.h>
#include <rccl/rccl.h>
#include <cstdint>
#include <cstdio>

#define HIP_CHECK(expr) do { hipError_t _e = (expr); if (_e != hipSuccess) { \
    fprintf(stderr, "HIP error %s at %s:%d\n", hipGetErrorString(_e), __FILE__, __LINE__); \
    return; } } while (0)

namespace {

typedef unsigned v4u __attribute__((ext_vector_type(4)));
typedef unsigned v2u __attribute__((ext_vector_type(2)));
typedef float f32x4 __attribute__((ext_vector_type(4)));
typedef short s16x4 __attribute__((ext_vector_type(4)));
typedef short s16x8 __attribute__((ext_vector_type(8)));
typedef __bf16 bf16x8 __attribute__((ext_vector_type(8)));
#define GAS __attribute__((address_space(1)))
#define LAS __attribute__((address_space(3)))

constexpr int NR = 8;
constexpr int KDIM = 4096, MSH = 512, NSH = 512;
constexpr int BK = 64, NSTEP = KDIM / BK;
constexpr int NSLOT = 5;
constexpr int ITEM = 8192;
constexpr int STREAM_BYTES = NSTEP * ITEM;
constexpr int NGEMM = 128;
#define AGQ_P 3
constexpr int PPH = AGQ_P;
constexpr int NPUSH_USED = 7 * PPH;
constexpr int NPUSH = (NPUSH_USED + 7) / 8 * 8;
constexpr int NWG = NPUSH + NGEMM;
constexpr int NTHR = 320;
#define AGQ_PF 3
constexpr int PF = AGQ_PF;
constexpr int NBUF = 4;
constexpr int STAGE = 32768;
constexpr unsigned long long DEADLINE_TICKS = 50000000ull;

constexpr size_t OFF_DATA = 0;
constexpr size_t OFF_FLAG = (size_t)NR * NSLOT * STREAM_BYTES;
constexpr size_t OFF_AMAX = OFF_FLAG + 16384;
constexpr size_t OFF_DONE = OFF_AMAX + 2 * NR * NGEMM * 8;
constexpr size_t OFF_CTR = OFF_DONE + 1024;
constexpr size_t OFF_TMO = OFF_CTR + 4096;
constexpr size_t OFF_DBG = OFF_TMO + 1024;
constexpr size_t WS_NEED = OFF_DBG + 65536;
static_assert(NWG <= 256 && NR * NSLOT * NSTEP * 4 <= 16384, "layout");

struct Params {
  const char* x;
  const char* w;
  char* ws[NR];
  char* out[NR];
  int rank;
};

#define SYS_RLX __ATOMIC_RELAXED, __HIP_MEMORY_SCOPE_SYSTEM
#define AGT_RLX __ATOMIC_RELAXED, __HIP_MEMORY_SCOPE_AGENT

#define STAMP(i) do { } while (0)

__device__ __forceinline__ char* pick_ptr(char* const (&tab)[NR], int idx) {
  char* r = tab[0];
#pragma unroll
  for (int i = 1; i < NR; ++i) r = (idx == i) ? tab[i] : r;
  return r;
}

__device__ __forceinline__ unsigned wave_max_u32(unsigned v) {
#pragma unroll
  for (int o = 32; o >= 1; o >>= 1) { unsigned u = __shfl_xor(v, o, 64); v = u > v ? u : v; }
  return v;
}

__global__ void __launch_bounds__(NTHR, 1) ag_gemm_q8(Params p) {
  __shared__ __attribute__((aligned(16))) char smem[NBUF * STAGE + 64];
  const int lane = threadIdx.x & 63;
  const int wave = __builtin_amdgcn_readfirstlane(threadIdx.x >> 6);
  const int me = p.rank, g = me >> 2, j = me & 3;
  char* const myws = pick_ptr(p.ws, me);
  GAS unsigned* const tmo = (GAS unsigned*)(myws + OFF_TMO);
  const int bid = blockIdx.x;
  GAS unsigned* const ctr = (GAS unsigned*)(myws + OFF_CTR) + bid;

  if (bid < NPUSH) {
    STAMP(0);
    if (bid < NPUSH_USED) {
      const int dst = (me + 1 + bid % 7) & 7;
      const int ph = bid / 7;
      const int slot = wave;
      const bool same = (dst >> 2) == g;
      const bool isw = !same && slot >= 3;
      unsigned voff, istride, tstride;
      if (isw) {
        voff = (unsigned)((slot - 3) * 32 + (lane >> 4)) * (NSH * 2) + (dst & 3) * 256 + (lane & 15) * 16;
        istride = 4 * NSH * 2; tstride = BK * NSH * 2;
      } else {
        voff = (unsigned)((same ? 0 : 320) + slot * 64 + (lane >> 3)) * (KDIM * 2) + (lane & 7) * 16;
        istride = 8 * KDIM * 2; tstride = BK * 2;
      }
      const char* sbase = isw ? p.w : p.x;
      char* dws = pick_ptr(p.ws, dst);
      const __amdgpu_buffer_rsrc_t srs = __builtin_amdgcn_make_buffer_rsrc((void*)sbase, 0, MSH * KDIM * 2, 0x00020000);
      const __amdgpu_buffer_rsrc_t drs = __builtin_amdgcn_make_buffer_rsrc((void*)(dws + OFF_DATA + (size_t)(me * NSLOT + slot) * STREAM_BYTES), 0, STREAM_BYTES, 0x00020000);
      GAS unsigned* flag = (GAS unsigned*)(dws + OFF_FLAG) + (me * NSLOT + slot) * NSTEP;
      v4u A[8], B[8];
#define PLOAD(R, t) _Pragma("unroll") for (int i = 0; i < 8; ++i) R[i] = __builtin_amdgcn_raw_buffer_load_b128(srs, voff + i * istride, (t) * tstride, 0)
#define PSTORE(R, t) _Pragma("unroll") for (int i = 0; i < 8; ++i) __builtin_amdgcn_raw_buffer_store_b128(R[i], drs, lane * 16 + i * 1024, (t) * ITEM, 17)
      const unsigned call = __hip_atomic_load(ctr, AGT_RLX) + 1;
      __builtin_amdgcn_sched_barrier(0);
      PLOAD(A, ph);
      __builtin_amdgcn_sched_barrier(0);
      PLOAD(B, ph + PPH);
      __builtin_amdgcn_sched_barrier(0);
      int prev = -1, t = ph;
      for (; t + PPH < NSTEP; t += 2 * PPH) {
        PSTORE(A, t);
        PLOAD(A, (t + 2 * PPH < NSTEP ? t + 2 * PPH : t));
        asm volatile("s_waitcnt vmcnt(16)" ::: "memory");
        if (prev >= 0 && lane == 0) __hip_atomic_store(flag + prev, call, SYS_RLX);
        PSTORE(B, t + PPH);
        PLOAD(B, (t + 3 * PPH < NSTEP ? t + 3 * PPH : t));
        asm volatile("s_waitcnt vmcnt(16)" ::: "memory");
        if (lane == 0) __hip_atomic_store(flag + t, call, SYS_RLX);
        prev = t + PPH;
      }
      if (t < NSTEP) {
        PSTORE(A, t);
        asm volatile("s_waitcnt vmcnt(8)" ::: "memory");
        if (lane == 0) __hip_atomic_store(flag + prev, call, SYS_RLX);
        prev = t;
      }
      asm volatile("s_waitcnt vmcnt(0)" ::: "memory");
      if (lane == 0) __hip_atomic_store(flag + prev, call, SYS_RLX);
      STAMP(1);
      __syncthreads();
      if (threadIdx.x == 0) __hip_atomic_store(ctr, call, AGT_RLX);
      if (bid == 0 && wave == 0) {
        GAS unsigned* done = (GAS unsigned*)(myws + OFF_DONE) + lane;
        const unsigned long long t0 = __builtin_amdgcn_s_memrealtime();
        while (true) {
          const unsigned v = __hip_atomic_load(done, SYS_RLX);
          if (__all(v >= call)) break;
          if (__any(__hip_atomic_load(tmo, AGT_RLX) == call) ||
              __builtin_amdgcn_s_memrealtime() - t0 > DEADLINE_TICKS) {
            if (lane == 0) {
              __hip_atomic_store(tmo, call, AGT_RLX);
              *(GAS signed char*)(pick_ptr(p.out, me) + (size_t)me * MSH * NSH) = -128;
            }
            break;
          }
          __builtin_amdgcn_s_sleep(4);
        }
        STAMP(2);
      }
    }
    return;
  }

  STAMP(0);
  const int gi = bid - NPUSH;
  const int xcd = gi & 7, idx = gi >> 3;
  const int rb = xcd >> 1;
  const int rt4 = idx & 3;
  const int ct = (idx >> 2) * 2 + (xcd & 1);
  const bool foreign = ct >= 4;
  const int wsrc = foreign ? (1 - g) * 4 + (ct - 4) : me;
  const int ocol = (foreign ? j : ct) * 128;
  const unsigned call = __builtin_amdgcn_readfirstlane(__hip_atomic_load(ctr, AGT_RLX)) + 1;
  LAS unsigned* const red = (LAS unsigned*)((LAS char*)smem + NBUF * STAGE);

  const size_t aoff = OFF_AMAX + (size_t)(call & 1) * (NR * NGEMM * 8);
  char* tw = p.ws[0];
#pragma unroll
  for (int i = 1; i < NR; ++i) tw = (lane == i) ? p.ws[i] : tw;
  tw += aoff + (size_t)(me * NGEMM + gi) * 8;
  char* ob = pick_ptr(p.out, wsrc);
  char* dn = pick_ptr(p.ws, wsrc) + OFF_DONE + (j * 16 + rb * 4 + rt4) * 4;
  asm volatile("" : "+v"(tw), "+s"(ob), "+s"(dn));

  f32x4 acc[4][4];
#pragma unroll
  for (int a = 0; a < 4; ++a)
#pragma unroll
    for (int b = 0; b < 4; ++b) acc[a][b] = f32x4{0.f, 0.f, 0.f, 0.f};

  if (wave == 4) {
    const int uu0 = rt4 * 2, uu1 = uu0 + 1;
    const int s0 = uu0 < 5 ? g * 4 + rb : (1 - g) * 4 + rb, l0 = uu0 < 5 ? uu0 : uu0 - 5;
    const int s1 = uu1 < 5 ? g * 4 + rb : (1 - g) * 4 + rb, l1 = uu1 < 5 ? uu1 : uu1 - 5;
    GAS unsigned* fl = (GAS unsigned*)(myws + OFF_FLAG) + lane;
    GAS unsigned* f0 = fl + (s0 * NSLOT + l0) * NSTEP;
    GAS unsigned* f1 = fl + (s1 * NSLOT + l1) * NSTEP;
    GAS unsigned* f2 = fl + (wsrc * NSLOT + 3) * NSTEP;
    GAS unsigned* f3 = f2 + NSTEP;
    const bool n0 = s0 != me, n1 = s1 != me;
    int known = (n0 || n1 || foreign) ? 0 : NSTEP;
    const unsigned long long t0 = __builtin_amdgcn_s_memrealtime();
    for (int n = 0; n < NSTEP + PF; ++n) {
      while (n < NSTEP && known <= n) {
        const unsigned a0 = __hip_atomic_load(f0, SYS_RLX), a1 = __hip_atomic_load(f1, SYS_RLX);
        const unsigned a2 = __hip_atomic_load(f2, SYS_RLX), a3 = __hip_atomic_load(f3, SYS_RLX);
        const unsigned tm = __hip_atomic_load(tmo, AGT_RLX);
        const bool ok = (!n0 || a0 >= call) && (!n1 || a1 >= call) && (!foreign || (a2 >= call && a3 >= call));
        const unsigned long long miss = ~__ballot(ok);
        known = miss ? __builtin_ctzll(miss) : NSTEP;
        if (known > n) break;
        if (__builtin_amdgcn_readfirstlane(tm) == call || __builtin_amdgcn_s_memrealtime() - t0 > DEADLINE_TICKS) {
          if (lane == 0) __hip_atomic_store(tmo, call, AGT_RLX);
          known = NSTEP;
          break;
        }
        __builtin_amdgcn_s_sleep(2);
      }
      if (n == NSTEP - 1) STAMP(1);
      asm volatile("s_barrier" ::: "memory");
    }
  } else {
    const int wm = wave >> 1, wn = wave & 1;
    const int uu = rt4 * 2 + wm;
    const int xs = uu < 5 ? g * 4 + rb : (1 - g) * 4 + rb, xl = uu < 5 ? uu : uu - 5;
    const bool xloc = xs == me;
    const char* xbase = xloc ? p.x + (size_t)uu * 64 * (KDIM * 2) : myws + OFF_DATA + (size_t)(xs * NSLOT + xl) * STREAM_BYTES;
    const unsigned xrs = xloc ? KDIM * 2 : 128, xts = xloc ? BK * 2 : ITEM;
    const char* wbase = foreign ? myws + OFF_DATA + (size_t)(wsrc * NSLOT + 3 + wm) * STREAM_BYTES
                                : p.w + (size_t)wm * 32 * (NSH * 2) + ct * 256;
    const unsigned wks = foreign ? 256 : NSH * 2, wts = foreign ? ITEM : BK * NSH * 2;
    unsigned xo[4], wo[4];
#pragma unroll
    for (int i = 0; i < 4; ++i) {
      const int r = wn * 32 + i * 8 + (lane >> 3);
      xo[i] = r * xrs + (((lane & 7) ^ ((r >> 1) & 7)) << 4);
      const int k = wn * 16 + i * 4 + (lane >> 4);
      const int f = (k & 3) | (((k >> 3) & 1) << 2);
      wo[i] = k * wks + (((lane & 15) ^ (f << 1)) << 4);
    }
    LAS char* const sm = (LAS char*)smem;
    const unsigned smbase = (unsigned)(size_t)sm + wave * 4096;
#define GLDS_(voff, sbase, inc) "global_load_lds_dwordx4 " voff ", " sbase " sc0 sc1\n\ts_add_u32 m0, m0, " inc "\n\ts_nop 0\n\t"
#define ISSUE(t) do { \
      const unsigned sb_ = smbase + ((t) & (NBUF - 1)) * STAGE; \
      const char* xb_ = xbase + (size_t)((unsigned)(t) * xts); \
      const char* wb_ = wbase + (size_t)((unsigned)(t) * wts); \
      unsigned keep_; \
      asm volatile("s_mov_b32 %0, m0\n\ts_mov_b32 m0, %1\n\ts_nop 3\n\t" \
                   GLDS_("%4", "%2", "0x400") GLDS_("%5", "%2", "0x400") GLDS_("%6", "%2", "0x400") GLDS_("%7", "%2", "0x3400") \
                   GLDS_("%8", "%3", "0x400") GLDS_("%9", "%3", "0x400") GLDS_("%10", "%3", "0x400") \
                   "global_load_lds_dwordx4 %11, %3 sc0 sc1\n\t" \
                   "s_mov_b32 m0, %0" \
                   : "=&s"(keep_) : "s"(sb_), "s"(xb_), "s"(wb_), "v"(xo[0]), "v"(xo[1]), "v"(xo[2]), "v"(xo[3]), \
                     "v"(wo[0]), "v"(wo[1]), "v"(wo[2]), "v"(wo[3]) : "memory", "scc"); \
    } while (0)
    const int fr = lane & 15, fg = lane >> 4;
    const unsigned xrd = (wm * 64 + fr) * 128 + ((fg ^ (fr >> 1)) << 4);
    const int q = (lane >> 2) & 3, pp = lane & 3;
    const unsigned wf = q | ((fg & 1) << 2);
    const unsigned wrd = 16384 + (fg * 8 + q) * 256 + ((((wn << 2) ^ wf) << 5) | (pp << 3));

#pragma unroll
    for (int n = 0; n < PF; ++n) {
      asm volatile("s_barrier" ::: "memory");
      ISSUE(n);
    }
    for (int t = 0; t < NSTEP; ++t) {
      const int rem = NSTEP - 1 - t;
      if (rem >= PF - 1) asm volatile("s_waitcnt vmcnt(%0) lgkmcnt(0)\n\ts_barrier" :: "i"(8 * (PF - 1)) : "memory");
      else if (PF == 3 && rem == 1) asm volatile("s_waitcnt vmcnt(8) lgkmcnt(0)\n\ts_barrier" ::: "memory");
      else asm volatile("s_waitcnt vmcnt(0) lgkmcnt(0)\n\ts_barrier" ::: "memory");
      if (t + PF < NSTEP) ISSUE(t + PF);
      LAS char* sb = sm + (t & (NBUF - 1)) * STAGE;
#pragma unroll
      for (int kk = 0; kk < 2; ++kk) {
        bf16x8 xf[4], wfr[4];
#pragma unroll
        for (int mt = 0; mt < 4; ++mt) xf[mt] = *(const LAS bf16x8*)(sb + ((xrd ^ (kk << 6)) + mt * 2048));
#pragma unroll
        for (int nt = 0; nt < 4; ++nt) {
          LAS char* a = sb + ((wrd ^ (nt << 5)) + kk * 8192);
          const s16x4 lo = __builtin_amdgcn_ds_read_tr16_b64_v4i16((LAS s16x4*)a);
          const s16x4 hi = __builtin_amdgcn_ds_read_tr16_b64_v4i16((LAS s16x4*)(a + 1024));
          wfr[nt] = __builtin_bit_cast(bf16x8, __builtin_shufflevector(lo, hi, 0, 1, 2, 3, 4, 5, 6, 7));
        }
#pragma unroll
        for (int mt = 0; mt < 4; ++mt)
#pragma unroll
          for (int nt = 0; nt < 4; ++nt)
            acc[mt][nt] = __builtin_amdgcn_mfma_f32_16x16x32_bf16(wfr[nt], xf[mt], acc[mt][nt], 0, 0, 0);
      }
    }
    STAMP(1);
    float m = 0.f;
#pragma unroll
    for (int a = 0; a < 4; ++a)
#pragma unroll
      for (int b = 0; b < 4; ++b)
#pragma unroll
        for (int r = 0; r < 4; ++r) { acc[a][b][r] = fmaxf(acc[a][b][r], 0.f); m = fmaxf(m, acc[a][b][r]); }
    const unsigned mu = wave_max_u32(__float_as_uint(m));
    if (lane == 0) red[wave] = mu;
  }
  __syncthreads();
  if (wave == 4) {
    unsigned mu = red[0];
#pragma unroll
    for (int i = 1; i < 4; ++i) mu = red[i] > mu ? red[i] : mu;
    if (lane < NR) __hip_atomic_store((GAS unsigned long long*)tw, ((unsigned long long)call << 32) | mu, SYS_RLX);
    if (lane == 0) __hip_atomic_store(ctr, call, AGT_RLX);
    STAMP(2);
  } else {
    const char* gp = myws + aoff + wave * 2048 + lane * 16;
    const unsigned long long t0 = __builtin_amdgcn_s_memrealtime();
    unsigned mu, bad = 0;
    while (true) {
      v4u a, b; unsigned tm;
      asm volatile("global_load_dwordx4 %0, %3, off sc0 sc1\n\t"
                   "global_load_dwordx4 %1, %3, off offset:1024 sc0 sc1\n\t"
                   "global_load_dword %2, %4, off sc1\n\t"
                   "s_waitcnt vmcnt(0)"
                   : "=&v"(a), "=&v"(b), "=&v"(tm) : "v"(gp), "v"(tmo) : "memory");
      const bool ok = a.y == call && a.w == call && b.y == call && b.w == call;
      const unsigned m0 = a.x > a.z ? a.x : a.z, m1 = b.x > b.z ? b.x : b.z;
      mu = m0 > m1 ? m0 : m1;
      if (__builtin_amdgcn_readfirstlane(tm) == call) { bad = 1; break; }
      if (__all(ok)) break;
      if (__builtin_amdgcn_s_memrealtime() - t0 > DEADLINE_TICKS) {
        if (lane == 0) __hip_atomic_store(tmo, call, AGT_RLX);
        bad = 1;
        break;
      }
      __builtin_amdgcn_s_sleep(1);
    }
    mu = wave_max_u32(mu);
    if (lane == 0) { red[4 + wave] = mu; red[8 + wave] = bad; }
    STAMP(2);
  }
  __syncthreads();
  if (wave < 4) {
    unsigned mu = red[4], bad = red[8];
#pragma unroll
    for (int i = 1; i < 4; ++i) { mu = red[4 + i] > mu ? red[4 + i] : mu; bad |= red[8 + i]; }
    const float amax = __uint_as_float(mu);
    const float scale = amax / 127.0f;
    const float inv = scale > 0.f ? 1.0f / scale : 0.f;
    const int wm = wave >> 1, wn = wave & 1;
    const int uu = rt4 * 2 + wm;
    const int xs = uu < 5 ? g * 4 + rb : (1 - g) * 4 + rb;
    const __amdgpu_buffer_rsrc_t ors = __builtin_amdgcn_make_buffer_rsrc((void*)ob, 0, NR * MSH * NSH, 0x00020000);
    const unsigned ooff = (unsigned)(xs * MSH + uu * 64 + (lane & 15)) * NSH + ocol + wn * 64 + (lane >> 4) * 16;
#pragma unroll
    for (int mt = 0; mt < 4; ++mt) {
      unsigned d[4];
#pragma unroll
      for (int nt = 0; nt < 4; ++nt) {
        unsigned v = 0;
#pragma unroll
        for (int r = 0; r < 4; ++r) {
          const float qf = fminf(__builtin_rintf(acc[mt][nt][r] * inv), 127.0f);
          v |= ((unsigned)(int)qf & 0xffu) << (8 * r);
        }
        d[nt] = bad ? 0x80808080u : v;
      }
      { auto r = __builtin_amdgcn_permlane32_swap(d[0], d[2], false, false); d[0] = r[0]; d[2] = r[1]; }
      { auto r = __builtin_amdgcn_permlane32_swap(d[1], d[3], false, false); d[1] = r[0]; d[3] = r[1]; }
      { auto r = __builtin_amdgcn_permlane16_swap(d[0], d[1], false, false); d[0] = r[0]; d[1] = r[1]; }
      { auto r = __builtin_amdgcn_permlane16_swap(d[2], d[3], false, false); d[2] = r[0]; d[3] = r[1]; }
      const v4u o = {d[0], d[1], d[2], d[3]};
      __builtin_amdgcn_raw_buffer_store_b128(o, ors, ooff + mt * 16 * NSH, 0, 17);
    }
    asm volatile("s_waitcnt vmcnt(0)" ::: "memory");
    STAMP(3);
  }
  if (foreign) {
    __syncthreads();
    if (threadIdx.x == 0) __hip_atomic_store((GAS unsigned*)dn, call, SYS_RLX);
  }
}

}

extern "C" void kernel_launch(int rank, int nranks, ncclComm_t comm,
                              hipStream_t stream,
                              void* const* d_in, const int* in_sizes, int n_in,
                              void* d_out, int out_size,
                              const int* in_dtypes, int out_dtype,
                              void* const* d_out_all,
                              void* const* d_ws_all, size_t ws_bytes) {
  (void)comm; (void)in_dtypes; (void)out_dtype; (void)d_out;
  if (nranks != NR || (unsigned)rank >= (unsigned)NR || n_in < 2 || in_sizes[0] != MSH * KDIM || in_sizes[1] != KDIM * NSH ||
      out_size != NR * MSH * NSH || ws_bytes < WS_NEED) {
    fprintf(stderr, "kernel_launch: unexpected problem shape\n");
    return;
  }
  Params p;
  p.x = (const char*)d_in[0];
  p.w = (const char*)d_in[1];
  for (int i = 0; i < NR; ++i) { p.ws[i] = (char*)d_ws_all[i]; p.out[i] = (char*)d_out_all[i]; }
  p.rank = rank;
  hipLaunchKernelGGL(ag_gemm_q8, dim3(NWG), dim3(NTHR), 0, stream, p);
  HIP_CHECK(hipGetLastError());
}
```

```cpp
#include <hip/hip_runtime.h>
#include <rccl/rccl.h>
#include <cstdint>
#include <cstdio>

#define HIP_CHECK(expr) do { hipError_t _e = (expr); if (_e != hipSuccess) { \
    fprintf(stderr, "HIP error %s at %s:%d\n", hipGetErrorString(_e), __FILE__, __LINE__); \
    return; } } while (0)

namespace {

typedef unsigned v4u __attribute__((ext_vector_type(4)));
typedef unsigned v2u __attribute__((ext_vector_type(2)));
typedef float f32x4 __attribute__((ext_vector_type(4)));
typedef short s16x4 __attribute__((ext_vector_type(4)));
typedef short s16x8 __attribute__((ext_vector_type(8)));
typedef __bf16 bf16x8 __attribute__((ext_vector_type(8)));
#define GAS __attribute__((address_space(1)))
#define LAS __attribute__((address_space(3)))

constexpr int NR = 8;
constexpr int KDIM = 4096, MSH = 512, NSH = 512;
constexpr int BK = 64, NSTEP = KDIM / BK;
constexpr int NSLOT = 5;
constexpr int ITEM = 8192;
constexpr int STREAM_BYTES = NSTEP * ITEM;
constexpr int NGEMM = 128;
#define AGQ_PW 8
#define AGQ_WW 1
#define AGQ_DYN 1
constexpr int PW = AGQ_PW;
constexpr int WW = AGQ_WW;
constexpr int PV = PW * WW;
constexpr int NITEM = NSTEP * NSLOT;
constexpr int NPUSH_USED = 7 * PW;
constexpr int NPUSH = (NPUSH_USED + 7) / 8 * 8;
constexpr int NWG = NPUSH + NGEMM;
constexpr int NTHR = 320;
static_assert(WW <= 4 && 3 * PV <= NITEM && MSH * KDIM == KDIM * NSH, "pusher shape");
#define AGQ_PF 2
constexpr int PF = AGQ_PF;
constexpr int NBUF = 4;
constexpr int STAGE = 32768;
constexpr unsigned long long DEADLINE_TICKS = 50000000ull;

constexpr size_t OFF_DATA = 0;
constexpr size_t OFF_FLAG = (size_t)NR * NSLOT * STREAM_BYTES;
constexpr size_t OFF_AMAX = OFF_FLAG + 16384;
constexpr size_t OFF_DONE = OFF_AMAX + 2 * NR * NGEMM * 8;
constexpr size_t OFF_CTR = OFF_DONE + 1024;
constexpr size_t OFF_TMO = OFF_CTR + 4096 * 2;
constexpr size_t OFF_TICK = OFF_TMO + 1024;
constexpr size_t OFF_DBG = OFF_TICK + 1024;
constexpr size_t WS_NEED = OFF_DBG + 65536;
static_assert(NWG <= 256 && NR * NSLOT * NSTEP * 4 <= 16384, "layout");

struct Params {
  const char* x;
  const char* w;
  char* ws[NR];
  char* out[NR];
  int rank;
};

#define SYS_RLX __ATOMIC_RELAXED, __HIP_MEMORY_SCOPE_SYSTEM
#define AGT_RLX __ATOMIC_RELAXED, __HIP_MEMORY_SCOPE_AGENT

#define STAMP(i) do { } while (0)

__device__ __forceinline__ char* pick_ptr(char* const (&tab)[NR], int idx) {
  char* r = tab[0];
#pragma unroll
  for (int i = 1; i < NR; ++i) r = (idx == i) ? tab[i] : r;
  return r;
}

__device__ __forceinline__ v4u make_rsrc(const void* base, unsigned bytes) {
  const unsigned long long a = (unsigned long long)base;
  return v4u{(unsigned)a, (unsigned)(a >> 32) & 0xffffu, bytes, 0x00020000u};
}

__device__ __forceinline__ unsigned wave_max_u32(unsigned v) {
#pragma unroll
  for (int o = 32; o >= 1; o >>= 1) { unsigned u = __shfl_xor(v, o, 64); v = u > v ? u : v; }
  return v;
}

__global__ void __launch_bounds__(NTHR, 1) ag_gemm_q8(Params p) {
  __shared__ __attribute__((aligned(16))) char smem[NBUF * STAGE + 64 + NSTEP * 4];
  const int lane = threadIdx.x & 63;
  const int wave = __builtin_amdgcn_readfirstlane(threadIdx.x >> 6);
  const int me = p.rank, g = me >> 2, j = me & 3;
  char* const myws = pick_ptr(p.ws, me);
  GAS unsigned* const tmo = (GAS unsigned*)(myws + OFF_TMO);
  const int bid = blockIdx.x;
  GAS unsigned* const ctr = (GAS unsigned*)(myws + OFF_CTR) + bid;

  if (bid < NPUSH) {
    STAMP(0);
    if (bid < NPUSH_USED && wave < WW) {
      const int dst = (me + 1 + bid % 7) & 7;
      const int v = (bid / 7) * WW + wave;
      const bool same = (dst >> 2) == g;
      const unsigned xlane = (unsigned)((same ? 0 : 320) + (lane >> 3)) * (KDIM * 2) + (lane & 7) * 16;
      const unsigned wlane = (unsigned)(lane >> 4) * (NSH * 2) + (dst & 3) * 256 + (lane & 15) * 16;
      char* dws = pick_ptr(p.ws, dst);
      const v4u drs = make_rsrc(dws + OFF_DATA + (size_t)me * NSLOT * STREAM_BYTES, NSLOT * STREAM_BYTES);
      const char* flag = dws + OFF_FLAG + me * NSLOT * NSTEP * 4;
      const char* tick = myws + OFF_TICK + (bid % 7) * 128;
      GAS unsigned* wctr = ctr + wave * 256;
      const unsigned vst0 = lane * 16, vst1 = lane * 16 + 4096, vzero = 0, vone = 1;
      v4u A[8], B[8];
#define POS(n) (((n) % NSLOT) * NSTEP + (n) / NSLOT)
#define PSRC(n) const int t_ = (n) / NSLOT, s_ = (n) % NSLOT; const bool w_ = !same && s_ >= 3; \
        const v4u rs_ = make_rsrc(w_ ? p.w : p.x, MSH * KDIM * 2); \
        const unsigned vo_ = w_ ? wlane : xlane; \
        unsigned so_ = w_ ? (t_ * BK + (s_ - 3) * 32) * (NSH * 2) : s_ * 64 * (KDIM * 2) + t_ * (BK * 2); \
        const unsigned is_ = w_ ? 4 * NSH * 2 : 8 * KDIM * 2
#define LD1(i) "buffer_load_dwordx4 %" #i ", %[vo], %[rs], %[so] offen\n\ts_add_u32 %[so], %[so], %[is]\n\t"
#define LD8 LD1(0) LD1(1) LD1(2) LD1(3) LD1(4) LD1(5) LD1(6) LD1(7)
#define ST8 "buffer_store_dwordx4 %0, %[v0], %[drs], %[dso] offen sc0 sc1\n\t" \
            "buffer_store_dwordx4 %1, %[v0], %[drs], %[dso] offen offset:1024 sc0 sc1\n\t" \
            "buffer_store_dwordx4 %2, %[v0], %[drs], %[dso] offen offset:2048 sc0 sc1\n\t" \
            "buffer_store_dwordx4 %3, %[v0], %[drs], %[dso] offen offset:3072 sc0 sc1\n\t" \
            "buffer_store_dwordx4 %4, %[v1], %[drs], %[dso] offen sc0 sc1\n\t" \
            "buffer_store_dwordx4 %5, %[v1], %[drs], %[dso] offen offset:1024 sc0 sc1\n\t" \
            "buffer_store_dwordx4 %6, %[v1], %[drs], %[dso] offen offset:2048 sc0 sc1\n\t" \
            "buffer_store_dwordx4 %7, %[v1], %[drs], %[dso] offen offset:3072 sc0 sc1\n\t"
#define PLOAD(R, n) do { PSRC(n); \
        asm volatile("s_nop 4\n\t" LD8 : "=&v"(R[0]), "=&v"(R[1]), "=&v"(R[2]), "=&v"(R[3]), "=&v"(R[4]), "=&v"(R[5]), "=&v"(R[6]), "=&v"(R[7]), \
                     [so] "+s"(so_) : [vo] "v"(vo_), [rs] "s"(rs_), [is] "s"(is_) : "memory", "scc"); } while (0)
      unsigned call;
      asm volatile("global_load_dword %0, %1, off sc1" : "=&v"(call) : "v"(wctr) : "memory");
      PLOAD(A, v);
      PLOAD(B, v + PV);
      asm volatile("s_waitcnt vmcnt(8)" : "+v"(call) :: "memory");
      call += 1;
      constexpr unsigned NDRAW = NITEM - 3 * PV + PV;
      const unsigned tbase = (call - 1) * NDRAW - 3 * PV;
      int prev = -1, na = v, nb = v + PV, nc = v + 2 * PV;
#define PROUND(R, cur) do { \
        const int ld_ = nc < NITEM ? nc : cur; \
        PSRC(ld_); \
        const unsigned dso_ = POS(cur) * ITEM, fo_ = POS(prev >= 0 ? prev : 0) * 4; \
        unsigned tk_; \
        asm volatile("s_cmp_lt_i32 %[nc], %[nitem]\n\ts_cselect_b64 exec, 1, 0\n\ts_nop 2\n\t" \
                     "global_atomic_add %[tk], %[vz], %[one], %[tick] sc0\n\t" \
                     "s_mov_b64 exec, -1\n\t" \
                     ST8 LD8 \
                     "s_waitcnt vmcnt(16)\n\t" \
                     "s_cmp_ge_i32 %[prev], 0\n\ts_cselect_b64 exec, 1, 0\n\t" \
                     "global_store_dword %[fo], %[call], %[flag] sc0 sc1\n\t" \
                     "s_mov_b64 exec, -1" \
                     : "+v"(R[0]), "+v"(R[1]), "+v"(R[2]), "+v"(R[3]), "+v"(R[4]), "+v"(R[5]), "+v"(R[6]), "+v"(R[7]), \
                       [so] "+s"(so_), [tk] "=&v"(tk_) \
                     : [vo] "v"(vo_), [rs] "s"(rs_), [is] "s"(is_), [v0] "v"(vst0), [v1] "v"(vst1), [drs] "s"(drs), [dso] "s"(dso_), \
                       [nc] "s"(nc), [nitem] "i"(AGQ_DYN ? NITEM : 0), [prev] "s"(prev), [vz] "v"(vzero), [one] "v"(vone), [tick] "s"(tick), [fo] "v"(fo_), \
                       [call] "v"(call), [flag] "s"(flag) \
                     : "memory", "scc"); \
        prev = cur; cur = nc; \
        const unsigned u_ = AGQ_DYN ? __builtin_amdgcn_readfirstlane(tk_ - tbase) : (unsigned)(nc + PV); \
        nc = (nc < NITEM && u_ < (unsigned)NITEM) ? (int)u_ : NITEM; \
      } while (0)
      while (true) {
        PROUND(A, na);
        if (nb >= NITEM) break;
        PROUND(B, nb);
        if (na >= NITEM) break;
      }
      asm volatile("s_waitcnt vmcnt(0)" ::: "memory");
      if (lane == 0) __hip_atomic_store((GAS unsigned*)flag + POS(prev), call, SYS_RLX);
      if (lane == 0) __hip_atomic_store(wctr, call, AGT_RLX);
      STAMP(1);
      if (bid == 0 && wave == 0) {
        GAS unsigned* done = (GAS unsigned*)(myws + OFF_DONE) + lane;
        const unsigned long long t0 = __builtin_amdgcn_s_memrealtime();
        while (true) {
          const unsigned v = __hip_atomic_load(done, SYS_RLX);
          if (__all(v >= call)) break;
          if (__any(__hip_atomic_load(tmo, AGT_RLX) == call) ||
              __builtin_amdgcn_s_memrealtime() - t0 > DEADLINE_TICKS) {
            if (lane == 0) {
              __hip_atomic_store(tmo, call, AGT_RLX);
              *(GAS signed char*)(pick_ptr(p.out, me) + (size_t)me * MSH * NSH) = -128;
            }
            break;
          }
          __builtin_amdgcn_s_sleep(4);
        }
        STAMP(2);
      }
    }
    return;
  }

  STAMP(0);
  const int gi = bid - NPUSH;
  const int xcd = gi & 7, idx = gi >> 3;
  const int rb = xcd >> 1;
  const int rt4 = idx & 3;
  const int ct = (idx >> 2) * 2 + (xcd & 1);
  const bool foreign = ct >= 4;
  const int wsrc = foreign ? (1 - g) * 4 + (ct - 4) : me;
  const int ocol = (foreign ? j : ct) * 128;
  const unsigned call = __builtin_amdgcn_readfirstlane(__hip_atomic_load(ctr, AGT_RLX)) + 1;
  LAS unsigned* const red = (LAS unsigned*)((LAS char*)smem + NBUF * STAGE);
  LAS unsigned* const sched = red + 16;

  const size_t aoff = OFF_AMAX + (size_t)(call & 1) * (NR * NGEMM * 8);
  char* tw = p.ws[0];
#pragma unroll
  for (int i = 1; i < NR; ++i) tw = (lane == i) ? p.ws[i] : tw;
  tw += aoff + (size_t)(me * NGEMM + gi) * 8;
  char* ob = pick_ptr(p.out, wsrc);
  char* dn = pick_ptr(p.ws, wsrc) + OFF_DONE + (j * 16 + rb * 4 + rt4) * 4;
  asm volatile("" : "+v"(tw));
  asm volatile("" : "+s"(ob), "+s"(dn));

  f32x4 acc[4][4];
#pragma unroll
  for (int a = 0; a < 4; ++a)
#pragma unroll
    for (int b = 0; b < 4; ++b) acc[a][b] = f32x4{0.f, 0.f, 0.f, 0.f};

  if (wave == 4) {
    const int uu0 = rt4 * 2, uu1 = uu0 + 1;
    const int s0 = uu0 < 5 ? g * 4 + rb : (1 - g) * 4 + rb, l0 = uu0 < 5 ? uu0 : uu0 - 5;
    const int s1 = uu1 < 5 ? g * 4 + rb : (1 - g) * 4 + rb, l1 = uu1 < 5 ? uu1 : uu1 - 5;
    GAS unsigned* fl = (GAS unsigned*)(myws + OFF_FLAG) + lane;
    GAS unsigned* f0 = fl + (s0 * NSLOT + l0) * NSTEP;
    GAS unsigned* f1 = fl + (s1 * NSLOT + l1) * NSTEP;
    GAS unsigned* f2 = fl + (wsrc * NSLOT + 3) * NSTEP;
    GAS unsigned* f3 = f2 + NSTEP;
    const bool n0 = s0 != me, n1 = s1 != me;
    unsigned long long avail = (n0 || n1 || foreign) ? 0ull : ~0ull, used = 0ull;
    const unsigned long long t0 = __builtin_amdgcn_s_memrealtime();
    for (int n = 0; n < NSTEP; ++n) {
      while (!avail) {
        const unsigned a0 = __hip_atomic_load(f0, SYS_RLX), a1 = __hip_atomic_load(f1, SYS_RLX);
        const unsigned a2 = __hip_atomic_load(f2, SYS_RLX), a3 = __hip_atomic_load(f3, SYS_RLX);
        const unsigned tm = __hip_atomic_load(tmo, AGT_RLX);
        const bool ok = (!n0 || a0 >= call) && (!n1 || a1 >= call) && (!foreign || (a2 >= call && a3 >= call));
        avail = __ballot(ok) & ~used;
        if (avail) break;
        if (__builtin_amdgcn_readfirstlane(tm) == call || __builtin_amdgcn_s_memrealtime() - t0 > DEADLINE_TICKS) {
          if (lane == 0) __hip_atomic_store(tmo, call, AGT_RLX);
          avail = ~used;
          break;
        }
        __builtin_amdgcn_s_sleep(2);
      }
      const int st = __builtin_ctzll(avail);
      avail &= avail - 1;
      used |= 1ull << st;
      if (lane == 0) sched[n] = st;
      if (n == NSTEP - 1) STAMP(1);
      asm volatile("s_waitcnt lgkmcnt(0)\n\ts_barrier" ::: "memory");
    }
#pragma unroll
    for (int n = 0; n < PF; ++n) asm volatile("s_barrier" ::: "memory");
  } else {
    const int wm = wave >> 1, wn = wave & 1;
    const int uu = rt4 * 2 + wm;
    const int xs = uu < 5 ? g * 4 + rb : (1 - g) * 4 + rb, xl = uu < 5 ? uu : uu - 5;
    const bool xloc = xs == me;
    const char* xbase = xloc ? p.x + (size_t)uu * 64 * (KDIM * 2) : myws + OFF_DATA + (size_t)(xs * NSLOT + xl) * STREAM_BYTES;
    const unsigned xrs = xloc ? KDIM * 2 : 128, xts = xloc ? BK * 2 : ITEM;
    const char* wbase = foreign ? myws + OFF_DATA + (size_t)(wsrc * NSLOT + 3 + wm) * STREAM_BYTES
                                : p.w + (size_t)wm * 32 * (NSH * 2) + ct * 256;
    const unsigned wks = foreign ? 256 : NSH * 2, wts = foreign ? ITEM : BK * NSH * 2;
    unsigned xo[4], wo[4];
#pragma unroll
    for (int i = 0; i < 4; ++i) {
      const int r = wn * 32 + i * 8 + (lane >> 3);
      xo[i] = r * xrs + (((lane & 7) ^ ((r >> 1) & 7)) << 4);
      const int k = wn * 16 + i * 4 + (lane >> 4);
      const int f = (k & 3) | (((k >> 3) & 1) << 2);
      wo[i] = k * wks + (((lane & 15) ^ (f << 1)) << 4);
    }
    LAS char* const sm = (LAS char*)smem;
    const unsigned smbase = (unsigned)(size_t)sm + wave * 4096;
#define GLDS(ldsoff, base, voff) asm volatile("s_mov_b32 m0, %0\n\ts_nop 0\n\tglobal_load_lds_dwordx4 %2, %1 sc0 sc1" \
                                              :: "s"(ldsoff), "s"(base), "v"(voff))
    const int fr = lane & 15, fg = lane >> 4;
    const unsigned xrd = (wm * 64 + fr) * 128 + ((fg ^ (fr >> 1)) << 4);
    const int q = (lane >> 2) & 3, pp = lane & 3;
    const unsigned wf = q | ((fg & 1) << 2);
    const unsigned wrd = 16384 + (fg * 8 + q) * 256 + ((((wn << 2) ^ wf) << 5) | (pp << 3));

#pragma unroll
    for (int n = 0; n < PF; ++n) {
      asm volatile("s_barrier" ::: "memory");
      const unsigned st = __builtin_amdgcn_readfirstlane(sched[n]);
      const char* xb = xbase + (size_t)(st * xts);
      const char* wb = wbase + (size_t)(st * wts);
      const unsigned lb = smbase + n * STAGE;
      asm volatile("s_nop 4");
#pragma unroll
      for (int i = 0; i < 4; ++i) GLDS(lb + i * 1024, xb, xo[i]);
#pragma unroll
      for (int i = 0; i < 4; ++i) GLDS(lb + 16384 + i * 1024, wb, wo[i]);
    }
#define READ_FRAGS(t) \
      LAS char* sb = sm + ((t) & (NBUF - 1)) * STAGE; \
      bf16x8 xf[2][4], wfr[2][4]; \
      _Pragma("unroll") for (int kk = 0; kk < 2; ++kk) { \
        _Pragma("unroll") for (int nt = 0; nt < 4; ++nt) { \
          LAS char* a = sb + ((wrd ^ (nt << 5)) + kk * 8192); \
          const s16x4 lo = __builtin_amdgcn_ds_read_tr16_b64_v4i16((LAS s16x4*)a); \
          const s16x4 hi = __builtin_amdgcn_ds_read_tr16_b64_v4i16((LAS s16x4*)(a + 1024)); \
          wfr[kk][nt] = __builtin_bit_cast(bf16x8, __builtin_shufflevector(lo, hi, 0, 1, 2, 3, 4, 5, 6, 7)); \
        } \
        _Pragma("unroll") for (int mt = 0; mt < 4; ++mt) xf[kk][mt] = *(const LAS bf16x8*)(sb + ((xrd ^ (kk << 6)) + mt * 2048)); \
      }
#define MMA4(kk, mt) _Pragma("unroll") for (int nt = 0; nt < 4; ++nt) \
      acc[mt][nt] = __builtin_amdgcn_mfma_f32_16x16x32_bf16(wfr[kk][nt], xf[kk][mt], acc[mt][nt], 0, 0, 0)
    for (int t = 0; t < NSTEP - PF; ++t) {
      asm volatile("s_waitcnt vmcnt(%0) lgkmcnt(0)\n\ts_barrier" :: "i"(8 * (PF - 1)) : "memory");
      const unsigned st = __builtin_amdgcn_readfirstlane(sched[t + PF]);
      READ_FRAGS(t);
      const char* xb = xbase + (size_t)(st * xts);
      const char* wb = wbase + (size_t)(st * wts);
      const unsigned lb = smbase + ((t + PF) & (NBUF - 1)) * STAGE;
      __builtin_amdgcn_sched_barrier(0);
      asm volatile("s_nop 4");
#pragma unroll
      for (int i = 0; i < 4; ++i) {
        GLDS(lb + i * 1024, xb, xo[i]);
        __builtin_amdgcn_sched_barrier(0);
        MMA4(0, i);
        __builtin_amdgcn_sched_barrier(0);
      }
#pragma unroll
      for (int i = 0; i < 4; ++i) {
        GLDS(lb + 16384 + i * 1024, wb, wo[i]);
        __builtin_amdgcn_sched_barrier(0);
        MMA4(1, i);
        __builtin_amdgcn_sched_barrier(0);
      }
    }
#pragma unroll
    for (int t = NSTEP - PF; t < NSTEP; ++t) {
      asm volatile("s_waitcnt vmcnt(%0) lgkmcnt(0)\n\ts_barrier" :: "i"(8 * (NSTEP - 1 - t)) : "memory");
      READ_FRAGS(t);
#pragma unroll
      for (int i = 0; i < 4; ++i) MMA4(0, i);
#pragma unroll
      for (int i = 0; i < 4; ++i) MMA4(1, i);
    }
    STAMP(1);
    float m = 0.f;
#pragma unroll
    for (int a = 0; a < 4; ++a)
#pragma unroll
      for (int b = 0; b < 4; ++b)
#pragma unroll
        for (int r = 0; r < 4; ++r) { acc[a][b][r] = fmaxf(acc[a][b][r], 0.f); m = fmaxf(m, acc[a][b][r]); }
    const unsigned mu = wave_max_u32(__float_as_uint(m));
    if (lane == 0) red[wave] = mu;
  }
  __syncthreads();
  if (wave == 4) {
    unsigned mu = red[0];
#pragma unroll
    for (int i = 1; i < 4; ++i) mu = red[i] > mu ? red[i] : mu;
    if (lane < NR) __hip_atomic_store((GAS unsigned long long*)tw, ((unsigned long long)call << 32) | mu, SYS_RLX);
    if (lane == 0) __hip_atomic_store(ctr, call, AGT_RLX);
    STAMP(2);
  } else {
    const char* gp = myws + aoff + wave * 2048 + lane * 16;
    const unsigned long long t0 = __builtin_amdgcn_s_memrealtime();
    unsigned mu, bad = 0;
    while (true) {
      v4u a, b; unsigned tm;
      asm volatile("global_load_dwordx4 %0, %3, off sc0 sc1\n\t"
                   "global_load_dwordx4 %1, %3, off offset:1024 sc0 sc1\n\t"
                   "global_load_dword %2, %4, off sc1\n\t"
                   "s_waitcnt vmcnt(0)"
                   : "=&v"(a), "=&v"(b), "=&v"(tm) : "v"(gp), "v"(tmo) : "memory");
      const bool ok = a.y == call && a.w == call && b.y == call && b.w == call;
      const unsigned m0 = a.x > a.z ? a.x : a.z, m1 = b.x > b.z ? b.x : b.z;
      mu = m0 > m1 ? m0 : m1;
      if (__builtin_amdgcn_readfirstlane(tm) == call) { bad = 1; break; }
      if (__all(ok)) break;
      if (__builtin_amdgcn_s_memrealtime() - t0 > DEADLINE_TICKS) {
        if (lane == 0) __hip_atomic_store(tmo, call, AGT_RLX);
        bad = 1;
        break;
      }
      __builtin_amdgcn_s_sleep(1);
    }
    mu = wave_max_u32(mu);
    if (lane == 0) { red[4 + wave] = mu; red[8 + wave] = bad; }
    STAMP(2);
  }
  __syncthreads();
  if (wave < 4) {
    unsigned mu = red[4], bad = red[8];
#pragma unroll
    for (int i = 1; i < 4; ++i) { mu = red[4 + i] > mu ? red[4 + i] : mu; bad |= red[8 + i]; }
    const float amax = __uint_as_float(mu);
    const float scale = amax / 127.0f;
    const float inv = scale > 0.f ? 1.0f / scale : 0.f;
    const int wm = wave >> 1, wn = wave & 1;
    const int uu = rt4 * 2 + wm;
    const int xs = uu < 5 ? g * 4 + rb : (1 - g) * 4 + rb;
    const __amdgpu_buffer_rsrc_t ors = __builtin_amdgcn_make_buffer_rsrc((void*)ob, 0, NR * MSH * NSH, 0x00020000);
    const unsigned ooff = (unsigned)(xs * MSH + uu * 64 + (lane & 15)) * NSH + ocol + wn * 64 + (lane >> 4) * 16;
#pragma unroll
    for (int mt = 0; mt < 4; ++mt) {
      unsigned d[4];
#pragma unroll
      for (int nt = 0; nt < 4; ++nt) {
        unsigned v = 0;
#pragma unroll
        for (int r = 0; r < 4; ++r) {
          const float qf = fminf(__builtin_rintf(acc[mt][nt][r] * inv), 127.0f);
          v |= ((unsigned)(int)qf & 0xffu) << (8 * r);
        }
        d[nt] = bad ? 0x80808080u : v;
      }
      { auto r = __builtin_amdgcn_permlane32_swap(d[0], d[2], false, false); d[0] = r[0]; d[2] = r[1]; }
      { auto r = __builtin_amdgcn_permlane32_swap(d[1], d[3], false, false); d[1] = r[0]; d[3] = r[1]; }
      { auto r = __builtin_amdgcn_permlane16_swap(d[0], d[1], false, false); d[0] = r[0]; d[1] = r[1]; }
      { auto r = __builtin_amdgcn_permlane16_swap(d[2], d[3], false, false); d[2] = r[0]; d[3] = r[1]; }
      const v4u o = {d[0], d[1], d[2], d[3]};
      __builtin_amdgcn_raw_buffer_store_b128(o, ors, ooff + mt * 16 * NSH, 0, 17);
    }
    asm volatile("s_waitcnt vmcnt(0)" ::: "memory");
    STAMP(3);
  }
  if (foreign) {
    __syncthreads();
    if (threadIdx.x == 0) __hip_atomic_store((GAS unsigned*)dn, call, SYS_RLX);
  }
}

}

extern "C" void kernel_launch(int rank, int nranks, ncclComm_t comm,
                              hipStream_t stream,
                              void* const* d_in, const int* in_sizes, int n_in,
                              void* d_out, int out_size,
                              const int* in_dtypes, int out_dtype,
                              void* const* d_out_all,
                              void* const* d_ws_all, size_t ws_bytes) {
  (void)comm; (void)in_dtypes; (void)out_dtype; (void)d_out;
  if (nranks != NR || (unsigned)rank >= (unsigned)NR || n_in < 2 || in_sizes[0] != MSH * KDIM || in_sizes[1] != KDIM * NSH ||
      out_size != NR * MSH * NSH || ws_bytes < WS_NEED) {
    fprintf(stderr, "kernel_launch: unexpected problem shape\n");
    return;
  }
  Params p;
  p.x = (const char*)d_in[0];
  p.w = (const char*)d_in[1];
  for (int i = 0; i < NR; ++i) { p.ws[i] = (char*)d_ws_all[i]; p.out[i] = (char*)d_out_all[i]; }
  p.rank = rank;
  hipLaunchKernelGGL(ag_gemm_q8, dim3(NWG), dim3(NTHR), 0, stream, p);
  HIP_CHECK(hipGetLastError());
}
```

```cpp
#include <hip/hip_runtime.h>
#include <rccl/rccl.h>
#include <cstdint>
#include <cstdio>

#define HIP_CHECK(expr) do { hipError_t _e = (expr); if (_e != hipSuccess) { \
    fprintf(stderr, "HIP error %s at %s:%d\n", hipGetErrorString(_e), __FILE__, __LINE__); \
    return; } } while (0)

namespace {

typedef unsigned v4u __attribute__((ext_vector_type(4)));
typedef unsigned v2u __attribute__((ext_vector_type(2)));
typedef float f32x4 __attribute__((ext_vector_type(4)));
typedef short s16x4 __attribute__((ext_vector_type(4)));
typedef __bf16 bf16x8 __attribute__((ext_vector_type(8)));
#define GAS __attribute__((address_space(1)))
#define LAS __attribute__((address_space(3)))

constexpr int NR = 8;
constexpr int KDIM = 4096, MSH = 512, NSH = 512;
constexpr int BK = 64, NSTEP = KDIM / BK;
constexpr int NSLOT = 5;
constexpr int ITEM = 8192;
constexpr int STREAM_BYTES = NSTEP * ITEM;
constexpr int NEARLY = 64, NLATE = 128;
constexpr int NGEMM = NEARLY + NLATE;
#define AGQ_PW 8
#define AGQ_WW 1
#define AGQ_DYN 1
#define AGQ_LAG 20
#define AGQ_INFL 2
#define AGQ_TH 432
#define AGQ_LPF 7
#define AGQ_POLL 20
constexpr int PW = AGQ_PW;
constexpr int WW = AGQ_WW;
constexpr int PV = PW * WW;
constexpr int NITEM = NSTEP * NSLOT;
constexpr int NPUSH_USED = 7 * PW;
constexpr int NPUSH = (NPUSH_USED + 7) / 8 * 8;
constexpr unsigned SPEC_MIN = AGQ_TH;
constexpr int LAG = AGQ_LAG;
constexpr int NWG = NPUSH + NGEMM;
constexpr int NTHR = 448;
static_assert(WW <= 4 && 3 * PV <= NITEM && MSH * KDIM == KDIM * NSH, "pusher shape");
constexpr int RING = 131072;
constexpr unsigned long long DEADLINE_TICKS = 50000000ull;

template <bool EARLY> struct Shape {
  static constexpr int NUN = EARLY ? 3 : 1;
  static constexpr int XB = NUN * 8192;
  static constexpr int WROW = EARLY ? 256 : 128;
  static constexpr int STAGE = XB + BK * WROW;
  static constexpr int NBUF = EARLY ? 3 : 8;
  static constexpr int PF = EARLY ? 2 : AGQ_LPF;
  static constexpr int NLD = STAGE / 2048;
  static constexpr int MT = EARLY ? 6 : 1;
  static_assert(NBUF * STAGE <= RING && PF < NBUF && PF * NLD < 64, "ring");
};

constexpr unsigned DATA_BYTES = NR * NSLOT * STREAM_BYTES;
constexpr size_t OFF_DATA = 0;
constexpr size_t OFF_FLAG = DATA_BYTES;
constexpr unsigned FLAG_BYTES = NR * NSLOT * NSTEP * 4;
constexpr size_t OFF_AMAX = OFF_FLAG + 16384;
constexpr size_t AMAX_BYTES = NR * NGEMM * 8;
constexpr size_t OFF_DONE = OFF_AMAX + 2 * AMAX_BYTES;
constexpr size_t OFF_CTR = OFF_DONE + 4096;
constexpr size_t OFF_TMO = OFF_CTR + 4096;
constexpr size_t OFF_TICK = OFF_TMO + 1024;
constexpr size_t OFF_RES = OFF_TICK + 1024;
constexpr size_t OFF_DBG = OFF_RES + 1024;
constexpr size_t WS_NEED = OFF_DBG + 131072;
static_assert(NWG <= 256 && FLAG_BYTES <= 16384 && AMAX_BYTES == 6 * 2048, "layout");

struct Params {
  const char* x;
  const char* w;
  char* ws[NR];
  char* out[NR];
  int rank;
};

#define SYS_RLX __ATOMIC_RELAXED, __HIP_MEMORY_SCOPE_SYSTEM
#define AGT_RLX __ATOMIC_RELAXED, __HIP_MEMORY_SCOPE_AGENT

#define STAMP(i) do { } while (0)
#define STAMPV(i, v) do { (void)(v); } while (0)

__device__ __forceinline__ char* pick_ptr(char* const (&tab)[NR], int idx) {
  char* r = tab[0];
#pragma unroll
  for (int i = 1; i < NR; ++i) r = (idx == i) ? tab[i] : r;
  return r;
}

__device__ __forceinline__ v4u make_rsrc(const void* base, unsigned bytes) {
  const unsigned long long a = (unsigned long long)base;
  return v4u{(unsigned)a, (unsigned)(a >> 32) & 0xffffu, bytes, 0x00020000u};
}

__device__ __forceinline__ unsigned umax(unsigned a, unsigned b) { return a > b ? a : b; }

__device__ __forceinline__ unsigned wave_max_u32(unsigned v) {
#define DPP_MAX(ctrl, rows) v = umax(v, __builtin_amdgcn_update_dpp(0u, v, ctrl, rows, 0xf, false));
  DPP_MAX(0x111, 0xf) DPP_MAX(0x112, 0xf) DPP_MAX(0x114, 0xf) DPP_MAX(0x118, 0xf)
  DPP_MAX(0x142, 0xa) DPP_MAX(0x143, 0xc)
  return __builtin_amdgcn_readlane(v, 63);
}

#define DMA16(ldsoff, voff, rsrc, soff) asm volatile("s_mov_b32 m0, %0\n\ts_nop 0\n\tbuffer_load_dwordx4 %1, %2, %3 offen sc0 sc1 lds" \
                                                     :: "s"(ldsoff), "v"(voff), "s"(rsrc), "s"(soff) : "memory")
#define DMA4(ldsoff, voff, rsrc) asm volatile("s_mov_b32 m0, %0\n\ts_nop 0\n\tbuffer_load_dword %1, %2, 0 offen sc0 sc1 lds" \
                                              :: "s"(ldsoff), "v"(voff), "s"(rsrc) : "memory")
#define BARRIER() asm volatile("s_waitcnt lgkmcnt(0)\n\ts_barrier" ::: "memory")

struct Job {
  int ur0, ur1, ur2;
  int wsrc, wct;
  int half;
  int nvalid;
  int gran, done, agg;
  __device__ __forceinline__ int unit(int u) const { return u == 0 ? ur0 : u == 1 ? ur1 : ur2; }
};
__device__ __forceinline__ int unit_stream(int ur) { return (ur >> 3) * NSLOT + ((ur & 7) < 5 ? (ur & 7) : (ur & 7) - 5); }

struct Ctl {
  unsigned red[4];
  v2u cmd;
  unsigned pad[10];
  unsigned sched[NSTEP];
  unsigned mir[5][NSTEP];
};

template <bool EARLY>
__device__ __forceinline__ void gemm_tile(const Params& p, const Job jb, char* const myws, LAS char* const sm,
                                          const unsigned call, const int lane, const int wave) {
  typedef Shape<EARLY> S;
  constexpr int PF = S::PF;
  const int me = p.rank;
  const bool foreign = jb.wsrc != me;
  volatile LAS Ctl* const ctl = (volatile LAS Ctl*)(sm + RING);
  GAS unsigned* const tmo = (GAS unsigned*)(myws + OFF_TMO);
  const unsigned smb = (unsigned)(size_t)sm;

  if (wave == 6) {
    const char* rp = myws + OFF_RES + jb.agg * 128;
    const v4u frs = make_rsrc(myws + OFF_FLAG, FLAG_BYTES);
    constexpr int NA = S::NUN + 2;
    unsigned fo[NA];
    unsigned need = 0;
#pragma unroll
    for (int a = 0; a < NA; ++a) {
      const int src = a < S::NUN ? jb.unit(a) >> 3 : jb.wsrc;
      fo[a] = (a < S::NUN ? unit_stream(jb.unit(a)) : jb.wsrc * NSLOT + 3 + a - S::NUN) * (NSTEP * 4) + lane * 4;
      need |= (src != me ? 1u : 0u) << a;
      ctl->mir[a][lane] = src != me ? 0u : call;
    }
    if (lane == 0) ctl->cmd = v2u{0u, 0u};
    asm volatile("s_waitcnt lgkmcnt(0)" ::: "memory");
    const unsigned mirb = smb + RING + (unsigned)offsetof(Ctl, mir);
    unsigned long long avail = need ? 0ull : ~0ull, used = 0ull;
    const unsigned long long t0 = __builtin_amdgcn_s_memrealtime();
    unsigned long long tl = t0 - AGQ_POLL;
    for (int n = 0; n < NSTEP; ++n) {
      for (unsigned spin = 1; need; ++spin) {
        const unsigned long long now = __builtin_amdgcn_s_memrealtime();
        unsigned long long all = ~0ull;
        unsigned left = 0;
#pragma unroll
        for (int a = 0; a < NA; ++a) {
          const unsigned long long b = __ballot(ctl->mir[a][lane] >= call);
          all &= b;
          left |= (b != ~0ull ? 1u : 0u) << a;
        }
        need = left;
        if (now - tl >= AGQ_POLL) {
          tl = now;
          asm volatile("s_nop 4");
#pragma unroll
          for (int a = 0; a < NA; ++a)
            if (need >> a & 1) DMA4(mirb + a * (NSTEP * 4), fo[a], frs);
        }
        avail = all & ~used;
        if (avail) break;
        if ((spin & 255) == 0 && (__builtin_amdgcn_readfirstlane(__hip_atomic_load(tmo, AGT_RLX)) == call || now - t0 > DEADLINE_TICKS)) {
          if (lane == 0) __hip_atomic_store(tmo, call, AGT_RLX);
          avail = ~used;
          need = 0;
          break;
        }
        __builtin_amdgcn_s_sleep(1);
      }
      const int st = __builtin_ctzll(avail);
      avail &= avail - 1;
      used |= 1ull << st;
      if (lane == 0) ctl->sched[n] = st;
      if (n == NSTEP - 1) STAMP(1);
      BARRIER();
    }
#pragma unroll
    for (int n = 0; n < PF - 1; ++n) BARRIER();
    v2u r;
    asm volatile("global_load_dwordx2 %0, %1, off sc1\n\ts_barrier\n\ts_waitcnt vmcnt(0)" : "=&v"(r) : "v"(rp) : "memory");
    unsigned last = 0;
    if (__builtin_amdgcn_readfirstlane(r.y) == 2 * call) {
      last = __builtin_amdgcn_readfirstlane(r.x);
      if (lane == 0) ctl->cmd = v2u{last, 1u};
    }
    BARRIER();
    while (true) {
      const unsigned long long rr = __hip_atomic_load((GAS unsigned long long*)rp, AGT_RLX);
      const unsigned tm = __hip_atomic_load(tmo, AGT_RLX);
      const unsigned tag = __builtin_amdgcn_readfirstlane((unsigned)(rr >> 32));
      const unsigned val = __builtin_amdgcn_readfirstlane((unsigned)rr);
      if (__builtin_amdgcn_readfirstlane(tm) == call || __builtin_amdgcn_s_memrealtime() - t0 > DEADLINE_TICKS) {
        if (lane == 0) { __hip_atomic_store(tmo, call, AGT_RLX); ctl->cmd = v2u{0u, 3u}; }
        break;
      }
      if (tag == 2 * call + 1) { if (lane == 0) ctl->cmd = v2u{val, 2u}; break; }
      if (tag == 2 * call && val > last) {
        if (lane == 0) ctl->cmd = v2u{val, 1u};
        last = val;
      }
      __builtin_amdgcn_s_sleep(1);
    }
    STAMP(3);
  } else if (wave >= 4) {
    const int l = wave - 4;
    char* tw = p.ws[0];
#pragma unroll
    for (int i = 1; i < NR; ++i) tw = (lane == i) ? p.ws[i] : tw;
    tw += OFF_AMAX + (size_t)(call & 1) * AMAX_BYTES + (size_t)jb.gran * 8;
    asm volatile("" : "+v"(tw));
    const v4u drs = make_rsrc(myws + OFF_DATA, DATA_BYTES);
    v4u xrs[S::NUN];
    unsigned xb[S::NUN], xt[S::NUN], xp[S::NUN], xv[S::NUN];
#pragma unroll
    for (int u = 0; u < S::NUN; ++u) {
      const bool loc = jb.unit(u) >> 3 == me;
      const unsigned rs = loc ? KDIM * 2 : 128;
      xrs[u] = loc ? make_rsrc(p.x, MSH * KDIM * 2) : drs;
      xb[u] = loc ? (jb.unit(u) & 7) * 64 * (KDIM * 2) : unit_stream(jb.unit(u)) * STREAM_BYTES;
      xt[u] = loc ? BK * 2 : ITEM;
      xp[u] = 16 * rs;
      xv[u] = (l * 8 + (lane >> 3)) * rs + (((lane & 7) ^ ((4 * l + (lane >> 4)) & 7)) << 4);
    }
    const v4u wrs = foreign ? drs : make_rsrc(p.w, KDIM * NSH * 2);
    const unsigned ks = foreign ? 256 : NSH * 2, wt = foreign ? ITEM : BK * NSH * 2, hs = foreign ? STREAM_BYTES : 32 * NSH * 2;
    const unsigned wb = (foreign ? (jb.wsrc * NSLOT + 3) * STREAM_BYTES : jb.wct * 256) + jb.half * 128;
    const unsigned wv = EARLY ? (8 * l + (lane >> 4)) * ks + (((lane & 15) ^ (((lane >> 4) | (l << 2)) << 1)) << 4)
                              : (8 * l + (lane >> 3)) * ks + (((lane & 7) ^ ((((lane >> 4) & 1) | (l << 1)) << 1)) << 4);
    unsigned lb = smb;
    for (int n = 0; n < NSTEP; ++n) {
      asm volatile("s_waitcnt vmcnt(%0)\n\ts_barrier" :: "i"(S::NLD * (PF - 1)) : "memory");
      const unsigned st = __builtin_amdgcn_readfirstlane(ctl->sched[n]) & (NSTEP - 1);
      asm volatile("s_nop 4");
#pragma unroll
      for (int u = 0; u < S::NUN; ++u)
#pragma unroll
        for (int m = 0; m < 4; ++m) DMA16(lb + u * 8192 + (2 * m + l) * 1024, xv[u], xrs[u], xb[u] + st * xt[u] + m * xp[u]);
      const unsigned ws0 = wb + st * wt;
#pragma unroll
      for (int m = 0; m < 4; ++m) {
        const unsigned so = ws0 + (m >> 1) * hs + (m & 1) * 16 * ks;
        if (EARLY) {
          DMA16(lb + S::XB + (4 * m + 2 * l) * 1024, wv, wrs, so);
          DMA16(lb + S::XB + (4 * m + 2 * l + 1) * 1024, wv, wrs, so + 4 * ks);
        } else {
          DMA16(lb + S::XB + (2 * m + l) * 1024, wv, wrs, so);
        }
      }
      lb = lb + S::STAGE == smb + S::NBUF * S::STAGE ? smb : lb + S::STAGE;
    }
#pragma unroll
    for (int n = 0; n < PF; ++n) asm volatile("s_waitcnt vmcnt(%0)\n\ts_barrier" :: "i"(S::NLD * (PF - 1 - n)) : "memory");
    if (l) return;
    BARRIER();
    const unsigned mu = umax(umax(ctl->red[0], ctl->red[1]), umax(ctl->red[2], ctl->red[3]));
    if (lane < NR) __hip_atomic_store((GAS unsigned long long*)tw, ((unsigned long long)call << 32) | mu, SYS_RLX);
    if (lane == 0) __hip_atomic_store((GAS unsigned*)(myws + OFF_CTR) + blockIdx.x, call, AGT_RLX);
    STAMP(2);
  } else {
    const int wm = EARLY ? wave >> 1 : wave, wn = EARLY ? wave & 1 : 0;
    char* ob = pick_ptr(p.out, jb.wsrc);
    char* dn = pick_ptr(p.ws, jb.wsrc) + OFF_DONE + (jb.done + wave) * 8;
    asm volatile("" : "+s"(ob), "+s"(dn));
    f32x4 acc[S::MT][4];
#pragma unroll
    for (int a = 0; a < S::MT; ++a)
#pragma unroll
      for (int b = 0; b < 4; ++b) acc[a][b] = f32x4{0.f, 0.f, 0.f, 0.f};
    const int fr = lane & 15, fg = lane >> 4;
    const unsigned xrd = (wm * S::MT * 16 + fr) * 128 + ((fg ^ (fr >> 1)) << 4);
    const int q = (lane >> 2) & 3, pp = lane & 3;
    const unsigned wf = EARLY ? (wn << 2) ^ (q | ((fg & 1) << 2)) : (q >> 1) | ((fg & 1) << 1);
    const unsigned wrd = S::XB + (fg * 8 + q) * S::WROW + ((wf << 5) | (pp << 3));
#pragma unroll
    for (int n = 0; n < PF; ++n) BARRIER();
    unsigned so = 0;
    for (int t = 0; t < NSTEP; ++t) {
      BARRIER();
      LAS char* sb = sm + so;
      bf16x8 xf[2][S::MT], wfr[2][4];
#pragma unroll
      for (int kk = 0; kk < 2; ++kk) {
#pragma unroll
        for (int nt = 0; nt < 4; ++nt) {
          LAS char* a = sb + ((wrd ^ (nt << 5)) + kk * 32 * S::WROW);
          const s16x4 lo = __builtin_amdgcn_ds_read_tr16_b64_v4i16((LAS s16x4*)a);
          const s16x4 hi = __builtin_amdgcn_ds_read_tr16_b64_v4i16((LAS s16x4*)(a + 4 * S::WROW));
          wfr[kk][nt] = __builtin_bit_cast(bf16x8, __builtin_shufflevector(lo, hi, 0, 1, 2, 3, 4, 5, 6, 7));
        }
#pragma unroll
        for (int mt = 0; mt < S::MT; ++mt) xf[kk][mt] = *(const LAS bf16x8*)(sb + ((xrd ^ (kk << 6)) + mt * 2048));
      }
      __builtin_amdgcn_sched_barrier(0);
#pragma unroll
      for (int kk = 0; kk < 2; ++kk)
#pragma unroll
        for (int mt = 0; mt < S::MT; ++mt)
#pragma unroll
          for (int nt = 0; nt < 4; ++nt)
            acc[mt][nt] = __builtin_amdgcn_mfma_f32_16x16x32_bf16(wfr[kk][nt], xf[kk][mt], acc[mt][nt], 0, 0, 0);
      so = so + S::STAGE == S::NBUF * S::STAGE ? 0 : so + S::STAGE;
    }
    STAMP(1);
    float m = 0.f;
#pragma unroll
    for (int a = 0; a < S::MT; ++a)
#pragma unroll
      for (int b = 0; b < 4; ++b)
#pragma unroll
        for (int r = 0; r < 4; ++r) m = fmaxf(m, acc[a][b][r]);
    const unsigned wmax = wave_max_u32(__float_as_uint(m));
    if (lane == 0) ctl->red[wave] = wmax;
    BARRIER();
    const unsigned own = __builtin_amdgcn_readfirstlane(umax(umax(ctl->red[0], ctl->red[1]), umax(ctl->red[2], ctl->red[3])));
    const __amdgpu_buffer_rsrc_t ors = __builtin_amdgcn_make_buffer_rsrc((void*)ob, 0, NR * MSH * NSH, 0x00020000);
    const unsigned ooff = (unsigned)fr * NSH + jb.wct * 128 + (jb.half + wn) * 64 + fg * 16;
    unsigned sent = 0xffffffffu, nsent = 0;
    const unsigned long long t0 = __builtin_amdgcn_s_memrealtime();
    while (true) {
      const v2u c = ctl->cmd;
      const unsigned mu = umax(__builtin_amdgcn_readfirstlane(c.x), own);
      unsigned state = __builtin_amdgcn_s_memrealtime() - t0 > 2 * DEADLINE_TICKS ? 3u : __builtin_amdgcn_readfirstlane(c.y);
      if (state != 0 && (mu != sent || state == 3)) {
        if (nsent == 0) STAMP(2);
        const float scale = __uint_as_float(mu) / 127.0f;
        const float inv = scale > 0.f ? 1.0f / scale : 0.f;
        bool stale = false;
        ++nsent;
#pragma unroll
        for (int mt = 0; mt < S::MT; ++mt) {
          const int tm = wm * S::MT + mt;
          if (tm >= jb.nvalid) break;
          if (mt >= AGQ_INFL && state == 1 && foreign) {
            asm volatile("s_waitcnt vmcnt(%0)" :: "i"(AGQ_INFL - 1) : "memory");
            const v2u c2 = ctl->cmd;
            state = __builtin_amdgcn_readfirstlane(c2.y);
            if (umax(__builtin_amdgcn_readfirstlane(c2.x), own) != mu || state == 3) { stale = true; break; }
          }
          unsigned d[4];
#pragma unroll
          for (int nt = 0; nt < 4; ++nt) {
            unsigned v = 0;
#pragma unroll
            for (int r = 0; r < 4; ++r) {
              const float qf = fminf(__builtin_rintf(fmaxf(acc[mt][nt][r], 0.f) * inv), 127.0f);
              v |= (unsigned)(int)qf << (8 * r);
            }
            d[nt] = state == 3 ? 0x80808080u : v;
          }
          { auto r = __builtin_amdgcn_permlane32_swap(d[0], d[2], false, false); d[0] = r[0]; d[2] = r[1]; }
          { auto r = __builtin_amdgcn_permlane32_swap(d[1], d[3], false, false); d[1] = r[0]; d[3] = r[1]; }
          { auto r = __builtin_amdgcn_permlane16_swap(d[0], d[1], false, false); d[0] = r[0]; d[1] = r[1]; }
          { auto r = __builtin_amdgcn_permlane16_swap(d[2], d[3], false, false); d[2] = r[0]; d[3] = r[1]; }
          const v4u o = {d[0], d[1], d[2], d[3]};
          __builtin_amdgcn_raw_buffer_store_b128(o, ors, ooff, (jb.unit(tm >> 2) * 64 + (tm & 3) * 16) * NSH, 17);
        }
        asm volatile("s_waitcnt vmcnt(0)" ::: "memory");
        if (stale) continue;
        if (foreign && lane == 0) __hip_atomic_store((GAS unsigned long long*)dn, ((unsigned long long)call << 32) | mu, SYS_RLX);
        sent = mu;
        STAMP(3);
      }
      if (state >= 2) break;
      __builtin_amdgcn_s_sleep(1);
    }
    STAMPV(4, nsent);
  }
}

__global__ void __launch_bounds__(NTHR, 1) ag_gemm_q8(Params p) {
  __shared__ __attribute__((aligned(16))) char smem[RING + sizeof(Ctl)];
  const int lane = threadIdx.x & 63;
  const int wave = __builtin_amdgcn_readfirstlane(threadIdx.x >> 6);
  const int me = p.rank, g = me >> 2, j = me & 3;
  char* const myws = pick_ptr(p.ws, me);
  GAS unsigned* const tmo = (GAS unsigned*)(myws + OFF_TMO);
  const int bid = blockIdx.x;
  GAS unsigned* const ctr = (GAS unsigned*)(myws + OFF_CTR) + bid;

  if (bid < NPUSH) {
    STAMP(0);
    if (bid < NPUSH_USED && wave < WW) {
      const int dst = (me + 1 + bid % 7) & 7;
      const int v = (bid / 7) * WW + wave;
      const bool same = (dst >> 2) == g;
      const unsigned xlane = (unsigned)((same ? 0 : 320) + (lane >> 3)) * (KDIM * 2) + (lane & 7) * 16;
      const unsigned wlane = (unsigned)(lane >> 4) * (NSH * 2) + (dst & 3) * 256 + (lane & 15) * 16;
      char* dws = pick_ptr(p.ws, dst);
      const v4u drs = make_rsrc(dws + OFF_DATA + (size_t)me * NSLOT * STREAM_BYTES, NSLOT * STREAM_BYTES);
      const char* flag = dws + OFF_FLAG + me * NSLOT * NSTEP * 4;
      const char* tick = myws + OFF_TICK + (bid % 7) * 128;
      GAS unsigned* wctr = ctr + wave * 256;
      const unsigned vst0 = lane * 16, vst1 = lane * 16 + 4096, vzero = 0, vone = 1;
      v4u A[8], B[8];
      auto POS = [same](int n) {
        const int m = n - 4 * LAG;
        const int e = m < 0 ? n & 3 : n >= NITEM - LAG ? 4 : m % NSLOT;
        const int t = m < 0 ? n >> 2 : n >= NITEM - LAG ? n - 4 * NSTEP : m / NSLOT + (e < 4 ? LAG : 0);
        return (same ? e : e < 4 ? e + 1 : 0) * NSTEP + t;
      };
#define PSRC(n) const int ps_ = POS(n), t_ = ps_ % NSTEP, s_ = ps_ / NSTEP; const bool w_ = !same && s_ >= 3; \
        const v4u rs_ = make_rsrc(w_ ? p.w : p.x, MSH * KDIM * 2); \
        const unsigned vo_ = w_ ? wlane : xlane; \
        unsigned so_ = w_ ? (t_ * BK + (s_ - 3) * 32) * (NSH * 2) : s_ * 64 * (KDIM * 2) + t_ * (BK * 2); \
        const unsigned is_ = w_ ? 4 * NSH * 2 : 8 * KDIM * 2
#define LD1(i) "buffer_load_dwordx4 %" #i ", %[vo], %[rs], %[so] offen\n\ts_add_u32 %[so], %[so], %[is]\n\t"
#define LD8 LD1(0) LD1(1) LD1(2) LD1(3) LD1(4) LD1(5) LD1(6) LD1(7)
#define ST8 "buffer_store_dwordx4 %0, %[v0], %[drs], %[dso] offen sc0 sc1\n\t" \
            "buffer_store_dwordx4 %1, %[v0], %[drs], %[dso] offen offset:1024 sc0 sc1\n\t" \
            "buffer_store_dwordx4 %2, %[v0], %[drs], %[dso] offen offset:2048 sc0 sc1\n\t" \
            "buffer_store_dwordx4 %3, %[v0], %[drs], %[dso] offen offset:3072 sc0 sc1\n\t" \
            "buffer_store_dwordx4 %4, %[v1], %[drs], %[dso] offen sc0 sc1\n\t" \
            "buffer_store_dwordx4 %5, %[v1], %[drs], %[dso] offen offset:1024 sc0 sc1\n\t" \
            "buffer_store_dwordx4 %6, %[v1], %[drs], %[dso] offen offset:2048 sc0 sc1\n\t" \
            "buffer_store_dwordx4 %7, %[v1], %[drs], %[dso] offen offset:3072 sc0 sc1\n\t"
#define PLOAD(R, n) do { PSRC(n); \
        asm volatile("s_nop 4\n\t" LD8 : "=&v"(R[0]), "=&v"(R[1]), "=&v"(R[2]), "=&v"(R[3]), "=&v"(R[4]), "=&v"(R[5]), "=&v"(R[6]), "=&v"(R[7]), \
                     [so] "+s"(so_) : [vo] "v"(vo_), [rs] "s"(rs_), [is] "s"(is_) : "memory", "scc"); } while (0)
      unsigned call;
      asm volatile("global_load_dword %0, %1, off sc1" : "=&v"(call) : "v"(wctr) : "memory");
      PLOAD(A, v);
      PLOAD(B, v + PV);
      asm volatile("s_waitcnt vmcnt(8)" : "+v"(call) :: "memory");
      call += 1;
      constexpr unsigned NDRAW = NITEM - 3 * PV + PV;
      const unsigned tbase = (call - 1) * NDRAW - 3 * PV;
      int prev = -1, na = v, nb = v + PV, nc = v + 2 * PV;
#define PROUND(R, cur) do { \
        const int ld_ = nc < NITEM ? nc : cur; \
        PSRC(ld_); \
        const unsigned dso_ = POS(cur) * ITEM, fo_ = POS(prev >= 0 ? prev : 0) * 4; \
        unsigned tk_; \
        asm volatile("s_cmp_lt_i32 %[nc], %[nitem]\n\ts_cselect_b64 exec, 1, 0\n\ts_nop 2\n\t" \
                     "global_atomic_add %[tk], %[vz], %[one], %[tick] sc0\n\t" \
                     "s_mov_b64 exec, -1\n\t" \
                     ST8 LD8 \
                     "s_waitcnt vmcnt(16)\n\t" \
                     "s_cmp_ge_i32 %[prev], 0\n\ts_cselect_b64 exec, 1, 0\n\t" \
                     "global_store_dword %[fo], %[call], %[flag] sc0 sc1\n\t" \
                     "s_mov_b64 exec, -1" \
                     : "+v"(R[0]), "+v"(R[1]), "+v"(R[2]), "+v"(R[3]), "+v"(R[4]), "+v"(R[5]), "+v"(R[6]), "+v"(R[7]), \
                       [so] "+s"(so_), [tk] "=&v"(tk_) \
                     : [vo] "v"(vo_), [rs] "s"(rs_), [is] "s"(is_), [v0] "v"(vst0), [v1] "v"(vst1), [drs] "s"(drs), [dso] "s"(dso_), \
                       [nc] "s"(nc), [nitem] "i"(AGQ_DYN ? NITEM : 0), [prev] "s"(prev), [vz] "v"(vzero), [one] "v"(vone), [tick] "s"(tick), [fo] "v"(fo_), \
                       [call] "v"(call), [flag] "s"(flag) \
                     : "memory", "scc"); \
        prev = cur; cur = nc; \
        const unsigned u_ = AGQ_DYN ? __builtin_amdgcn_readfirstlane(tk_ - tbase) : (unsigned)(nc + PV); \
        nc = (nc < NITEM && u_ < (unsigned)NITEM) ? (int)u_ : NITEM; \
      } while (0)
      int nround = 0;
      while (true) {
        PROUND(A, na);
        ++nround;
        if (nb >= NITEM) break;
        PROUND(B, nb);
        ++nround;
        if (na >= NITEM) break;
      }
      asm volatile("s_waitcnt vmcnt(0)" ::: "memory");
      if (lane == 0) __hip_atomic_store((GAS unsigned*)flag + POS(prev), call, SYS_RLX);
      if (lane == 0) __hip_atomic_store(wctr, call, AGT_RLX);
      STAMP(1);
      STAMPV(4, nround);
      STAMPV(5, __builtin_amdgcn_s_getreg(20 | (15 << 11)));
    }
    return;
  }

  STAMP(0);
  const unsigned call = __builtin_amdgcn_readfirstlane(__hip_atomic_load(ctr, AGT_RLX)) + 1;
  const int gi = bid - NPUSH;
  const int rb = (gi & 7) >> 1, idx = gi >> 3;
  bool early = idx < 8;
  const int sel = idx & 1;
  const int half = early ? 0 : (idx >> 1) & 1;
  const int ct = (early ? idx >> 1 : (idx - 8) >> 2) * 2 + (gi & 1);
  const int sa = g * 4 + rb, sb = (1 - g) * 4 + rb;
  const bool diag = sel == 1 && ct == 4 + rb;
  Job jb;
  jb.ur0 = early ? sa * 8 + 3 * sel : sel ? sb * 8 + 5 : sa * 8 + 4;
  jb.ur1 = !sel ? sa * 8 + 1 : diag ? sa * 8 + 3 : sb * 8 + 6;
  jb.ur2 = !sel ? sa * 8 + 2 : diag ? sa * 8 + 3 : sb * 8 + 7;
  jb.wsrc = ct >= 4 ? (1 - g) * 4 + (ct - 4) : me;
  jb.wct = ct >= 4 ? j : ct;
  jb.half = half;
  jb.nvalid = diag ? 4 : 12;
  jb.gran = early ? me * NEARLY + gi : NR * NEARLY + me * NLATE + gi - NEARLY;
  jb.done = j * 128 + (early ? (rb * 2 + sel) * 4 : 32 + ((rb * 2 + sel) * 2 + half) * 4);
  jb.agg = rb;

  if (!early && diag && half == 1) {
    volatile LAS v2u* const part = (volatile LAS v2u*)((LAS char*)smem + RING);
    if (wave < 6 && lane == 0) part[wave] = v2u{0u, 0u};
    __syncthreads();
    const size_t aoff = OFF_AMAX + (size_t)(call & 1) * AMAX_BYTES;
    const unsigned long long t0 = __builtin_amdgcn_s_memrealtime();
    if (wave < 6) {
      const char* gp = myws + aoff + wave * 2048 + lane * 16;
      unsigned mu = 0, cnt = 0, last = 0;
      while (true) {
        unsigned tm = 0;
        if (cnt < 4) {
          v4u a, b;
          asm volatile("global_load_dwordx4 %0, %3, off sc0 sc1\n\t"
                       "global_load_dwordx4 %1, %3, off offset:1024 sc0 sc1\n\t"
                       "global_load_dword %2, %4, off sc1\n\t"
                       "s_waitcnt vmcnt(0)"
                       : "=&v"(a), "=&v"(b), "=&v"(tm) : "v"(gp), "v"(tmo) : "memory");
          cnt = (a.y == call) + (a.w == call) + (b.y == call) + (b.w == call);
          mu = umax(umax(a.y == call ? a.x : 0u, a.w == call ? a.z : 0u), umax(b.y == call ? b.x : 0u, b.w == call ? b.z : 0u));
        }
        const unsigned tot = __builtin_popcountll(__ballot(cnt & 1)) + 2 * __builtin_popcountll(__ballot(cnt & 2)) +
                             4 * __builtin_popcountll(__ballot(cnt & 4));
        if (tot != last) {
          const unsigned m = wave_max_u32(mu);
          if (lane == 0) part[wave] = v2u{m, tot};
          last = tot;
        }
        if (tot == 256 || __any(tm == call) || __builtin_amdgcn_s_memrealtime() - t0 > DEADLINE_TICKS) break;
        __builtin_amdgcn_s_sleep(1);
      }
      return;
    }
    char* tw = p.ws[0];
#pragma unroll
    for (int i = 1; i < NR; ++i) tw = (lane == i) ? p.ws[i] : tw;
    if (lane < NR) __hip_atomic_store((GAS unsigned long long*)(tw + aoff + (size_t)jb.gran * 8), (unsigned long long)call << 32, SYS_RLX);
    GAS unsigned long long* rp = (GAS unsigned long long*)(myws + OFF_RES + rb * 128);
    unsigned mx = 0, pub = 0;
    bool bad = false;
    for (unsigned it = 0;; ++it) {
      const v2u p0 = part[0], p1 = part[1], p2 = part[2], p3 = part[3], p4 = part[4], p5 = part[5];
      mx = __builtin_amdgcn_readfirstlane(umax(umax(umax(p0.x, p1.x), umax(p2.x, p3.x)), umax(p4.x, p5.x)));
      const unsigned ne = __builtin_amdgcn_readfirstlane(p0.y + p1.y);
      const unsigned nl = __builtin_amdgcn_readfirstlane(p2.y + p3.y + p4.y + p5.y);
      if (ne + nl == NR * NGEMM) break;
      if (ne >= SPEC_MIN && mx != pub) {
        if (pub == 0) STAMP(3);
        if (lane == 0) __hip_atomic_store(rp, ((unsigned long long)(2 * call) << 32) | mx, AGT_RLX);
        pub = mx;
      }
      if ((it & 63) == 63 && (__builtin_amdgcn_readfirstlane(__hip_atomic_load(tmo, AGT_RLX)) == call ||
                              __builtin_amdgcn_s_memrealtime() - t0 > DEADLINE_TICKS)) { bad = true; break; }
      __builtin_amdgcn_s_sleep(1);
    }
    if (!bad && lane == 0) {
      asm volatile("s_waitcnt vmcnt(0)" ::: "memory");
      __hip_atomic_store(rp, ((unsigned long long)(2 * call + 1) << 32) | mx, AGT_RLX);
    }
    if (lane == 0) __hip_atomic_store(ctr, call, AGT_RLX);
    STAMP(1);
    const char* gp = myws + OFF_DONE + rb * 1024 + lane * 16;
    const bool idle = lane >= 48 || (lane >> 2) == 4 + j * 2 + 1;
    while (!bad) {
      v4u a; unsigned tm;
      asm volatile("global_load_dwordx4 %0, %2, off sc0 sc1\n\t"
                   "global_load_dword %1, %3, off sc1\n\t"
                   "s_waitcnt vmcnt(0)"
                   : "=&v"(a), "=&v"(tm) : "v"(gp), "v"(tmo) : "memory");
      if (__all(idle || (a.y == call && a.w == call && a.x == mx && a.z == mx))) break;
      if (__any(tm == call) || __builtin_amdgcn_s_memrealtime() - t0 > DEADLINE_TICKS) bad = true;
      __builtin_amdgcn_s_sleep(1);
    }
    STAMP(2);
    if (bad && lane == 0) {
      __hip_atomic_store(tmo, call, AGT_RLX);
      *(GAS signed char*)(pick_ptr(p.out, me) + (size_t)me * MSH * NSH) = -128;
    }
    return;
  }
  if (!early && diag) {
    early = true;
    jb.ur0 = me * 8 + 5; jb.ur1 = me * 8 + 6; jb.ur2 = me * 8 + 7;
    jb.wsrc = me; jb.wct = rb; jb.nvalid = 12;
  }
  if (early) gemm_tile<true>(p, jb, myws, (LAS char*)smem, call, lane, wave);
  else gemm_tile<false>(p, jb, myws, (LAS char*)smem, call, lane, wave);
}

}

extern "C" void kernel_launch(int rank, int nranks, ncclComm_t comm,
                              hipStream_t stream,
                              void* const* d_in, const int* in_sizes, int n_in,
                              void* d_out, int out_size,
                              const int* in_dtypes, int out_dtype,
                              void* const* d_out_all,
                              void* const* d_ws_all, size_t ws_bytes) {
  (void)comm; (void)in_dtypes; (void)out_dtype; (void)d_out;
  if (nranks != NR || (unsigned)rank >= (unsigned)NR || n_in < 2 || in_sizes[0] != MSH * KDIM || in_sizes[1] != KDIM * NSH ||
      out_size != NR * MSH * NSH || ws_bytes < WS_NEED) {
    fprintf(stderr, "kernel_launch: unexpected problem shape\n");
    return;
  }
  Params p;
  p.x = (const char*)d_in[0];
  p.w = (const char*)d_in[1];
  for (int i = 0; i < NR; ++i) { p.ws[i] = (char*)d_ws_all[i]; p.out[i] = (char*)d_out_all[i]; }
  p.rank = rank;
  hipLaunchKernelGGL(ag_gemm_q8, dim3(NWG), dim3(NTHR), 0, stream, p);
  HIP_CHECK(hipGetLastError());
}
```

```cpp
#include <hip/hip_runtime.h>
#include <rccl/rccl.h>
#include <cstdint>
#include <cstdio>

#define HIP_CHECK(expr) do { hipError_t _e = (expr); if (_e != hipSuccess) { \
    fprintf(stderr, "HIP error %s at %s:%d\n", hipGetErrorString(_e), __FILE__, __LINE__); \
    return; } } while (0)

namespace {

typedef unsigned v4u __attribute__((ext_vector_type(4)));
typedef unsigned v2u __attribute__((ext_vector_type(2)));
typedef unsigned v16u __attribute__((ext_vector_type(16)));
typedef float f32x4 __attribute__((ext_vector_type(4)));
typedef float f32x32 __attribute__((ext_vector_type(32)));
typedef short s16x4 __attribute__((ext_vector_type(4)));
typedef __bf16 bf16x8 __attribute__((ext_vector_type(8)));
#define GAS __attribute__((address_space(1)))
#define LAS __attribute__((address_space(3)))

constexpr int NR = 8;
constexpr int KDIM = 4096, MSH = 512, NSH = 512;
constexpr int BK = 64, NSTEP = KDIM / BK;
constexpr int NSLOT = 5;
constexpr int ITEM = 8192;
constexpr int STREAM_BYTES = NSTEP * ITEM;
constexpr int NEARLY = 64, NLATE = 128;
constexpr int NGEMM = NEARLY + NLATE;
#define AGQ_PW 4
#define AGQ_WW 2
#define AGQ_XI 0
#define AGQ_XC 0
#define AGQ_GMAP 0
#define AGQ_L2P 0x75416320u
#define AGQ_LAG 40
#define AGQ_INFL 2
#define AGQ_TH 432
#define AGQ_POLL 20
#define AGQ_NSPEC 8
#define AGQ_WIN 16
#define AGQ_EWIN 6
#define AGQ_LK 6
#define AGQ_EK 1
#define AGQ_CLRT 400
#define AGQ_CLRN 1
#define AGQ_CLRD 36
#define AGQ_NSWEEP 24
#define AGQ_WREG 1
#define AGQ_PREF 1
#define AGQ_GATE 0
#define AGQ_GT 160
#define AGQ_PRIO 2
constexpr int PW = AGQ_PW;
constexpr int WW = AGQ_WW;
constexpr int PV = PW * WW;
constexpr int NITEM = NSTEP * NSLOT;
constexpr int NPUSH = 8 * PW;
constexpr unsigned NDRAW = NITEM - 4 * PV + PV;
constexpr unsigned L2P = AGQ_L2P;
#define PUSH_DST1(me, b) (((b) & 4) ? (((me) & 4) ^ 4) + (((b) ^ AGQ_XC) & 3) : ((me) & 4) + (((b) ^ AGQ_XI) & 3))
#define PUSH_DST(me, b) (PUSH_DST1(me, b) == (me) ? -1 : PUSH_DST1(me, b))
constexpr unsigned SPEC_MIN = AGQ_TH;
constexpr int LAG = AGQ_LAG;
constexpr int NSPEC = AGQ_NSPEC;
constexpr int NSWEEP = AGQ_NSWEEP;
constexpr int NWG = NPUSH + NGEMM + NSWEEP;
constexpr int NTHR = 512;
static_assert(WW <= 4 && 4 * PV <= NITEM && PV <= 8 && MSH * KDIM == KDIM * NSH, "pusher shape");
constexpr int RING = 19 * ITEM;
constexpr unsigned long long DEADLINE_TICKS = 50000000ull;

constexpr int E_XB = 3 * 8192, E_STAGE = E_XB + BK * 256, E_NBUF = 3, E_PF = 2, E_NLD = 14;
constexpr int NWORK = 7;
__device__ __forceinline__ int work_lds(int w) { return (w < 5 ? 3 * w : 2 * w + 5) * ITEM; }
static_assert(E_NBUF * E_STAGE <= RING && E_PF < E_NBUF && E_PF * E_NLD < 64, "ring");

constexpr unsigned DATA_BYTES = NR * NSLOT * STREAM_BYTES;
constexpr size_t OFF_DATA = 0;
constexpr size_t OFF_FLAG = 2 * (size_t)DATA_BYTES;
constexpr unsigned FLAG_BYTES = NR * NSLOT * NSTEP * 4;
constexpr size_t OFF_AMAX = OFF_FLAG + 16384;
constexpr size_t AMAX_BYTES = NR * NGEMM * 8;
constexpr size_t OFF_DONE = OFF_AMAX + 2 * AMAX_BYTES;
constexpr size_t OFF_CTR = OFF_DONE + 4096;
constexpr size_t OFF_TMO = OFF_CTR + 4096;
constexpr size_t OFF_TICK = OFF_TMO + 1024;
constexpr size_t OFF_RES = OFF_TICK + 1024;
constexpr size_t OFF_DBG = OFF_RES + 1024;
constexpr size_t WS_NEED = OFF_DBG + 131072 + 256 * 64 * 32;
static_assert(NWG <= 256 && FLAG_BYTES <= 16384 && AMAX_BYTES == 6 * 2048 && NSWEEP <= 64 && NSPEC >= 1 && 5 * NSPEC <= 64, "layout");

struct Params {
  const char* x;
  const char* w;
  char* myws;
  int rank;
  char* pd[NR];
  char* ws[NR];
  char* out[NR];
};

#define SYS_RLX __ATOMIC_RELAXED, __HIP_MEMORY_SCOPE_SYSTEM
#define AGT_RLX __ATOMIC_RELAXED, __HIP_MEMORY_SCOPE_AGENT

#define STAMP(i) do { } while (0)
#define STAMPV(i, v) do { (void)(v); } while (0)

#define TRACE(st, i) do { } while (0)

__device__ __forceinline__ char* pick_ptr(char* const (&tab)[NR], int idx) {
  char* r = tab[0];
#pragma unroll
  for (int i = 1; i < NR; ++i) r = (idx == i) ? tab[i] : r;
  return r;
}

__device__ __forceinline__ v4u make_rsrc(const void* base, unsigned bytes) {
  const unsigned long long a = (unsigned long long)base;
  return v4u{(unsigned)a, (unsigned)(a >> 32) & 0xffffu, bytes, 0x00020000u};
}

__device__ __forceinline__ unsigned umax(unsigned a, unsigned b) { return a > b ? a : b; }

__device__ __forceinline__ unsigned wave_max_u32(unsigned v) {
#define DPP_MAX(ctrl, rows) v = umax(v, __builtin_amdgcn_update_dpp(0u, v, ctrl, rows, 0xf, false));
  DPP_MAX(0x111, 0xf) DPP_MAX(0x112, 0xf) DPP_MAX(0x114, 0xf) DPP_MAX(0x118, 0xf)
  DPP_MAX(0x142, 0xa) DPP_MAX(0x143, 0xc)
  return __builtin_amdgcn_readlane(v, 63);
}

#define DMA16(ldsoff, voff, rsrc, soff) asm volatile("s_mov_b32 m0, %0\n\ts_nop 0\n\tbuffer_load_dwordx4 %1, %2, %3 offen sc0 sc1 lds" \
                                                     :: "s"(ldsoff), "v"(voff), "s"(rsrc), "s"(soff) : "memory")
#define DMA4(ldsoff, voff, rsrc) asm volatile("s_mov_b32 m0, %0\n\ts_nop 0\n\tbuffer_load_dword %1, %2, 0 offen sc0 sc1 lds" \
                                              :: "s"(ldsoff), "v"(voff), "s"(rsrc) : "memory")
#define BARRIER() asm volatile("s_waitcnt lgkmcnt(0)\n\ts_barrier" ::: "memory")

struct Job {
  int ur0, ur1, ur2;
  int wsrc, wct;
  int half;
  int nvalid;
  int gran, done, agg;
  __device__ __forceinline__ int unit(int u) const { return u == 0 ? ur0 : u == 1 ? ur1 : ur2; }
};
__device__ __forceinline__ int unit_row(int ur) { return (L2P >> (ur >> 3) * 4 & 7) * MSH + (ur & 7) * 64; }
__device__ __forceinline__ int unit_stream(int ur) { return (ur >> 3) * NSLOT + ((ur & 7) < 5 ? (ur & 7) : (ur & 7) - 5); }

struct Ctl {
  v2u res;
  v2u gate;
  unsigned tmax, nmax, pad2[2];
  unsigned fin[8];
  unsigned sched[NSTEP];
  unsigned mir[5][NSTEP];
  unsigned prb[NSTEP];
};
#define WG_RLX __ATOMIC_RELAXED, __HIP_MEMORY_SCOPE_WORKGROUP

#define LD2(i, k, sc) "buffer_load_dwordx4 %" #i ", %[vo], %[rs], %[so] offen" sc "\n\ts_add_u32 %[so], %[so], %[i0]\n\t" \
                      "buffer_load_dwordx4 %" #k ", %[vo], %[rs], %[so] offen" sc "\n\ts_add_u32 %[so], %[so], %[i1]\n\t"
#define LD8(sc) LD2(0, 1, sc) LD2(2, 3, sc) LD2(4, 5, sc) LD2(6, 7, sc)

#define VR(b, i) "v[" #b "+4*(" #i "):" #b "+4*(" #i ")+3]"
#define MN(b, i) "v_min3_u32 %[mn], %[mn], v[" #b "+" #i "], v[" #b "+" #i "+1]\n\t"

#define PDMA(i, rs, so, off) "buffer_load_dwordx4 v[80+" #i "], %[" rs "], %[" so "] offen offset:" #off " sc0 sc1 lds\n\t"
#define M0ADD "s_add_u32 m0, m0, 4096\n\ts_nop 0\n\t"

template <bool EARLY>
__device__ __forceinline__ void poll_wave(const Params& p, const Job jb, char* const myws, LAS char* const sm,
                                          const unsigned call, const int lane, const int wave) {
  constexpr int NUN = EARLY ? 3 : 1, NA = NUN + 2, S0 = NSTEP - NSPEC;
  const int me = p.rank;
  volatile LAS Ctl* const ctl = (volatile LAS Ctl*)(sm + RING);
  GAS unsigned* const tmo = (GAS unsigned*)(myws + OFF_TMO);
  const unsigned ctb = (unsigned)(size_t)sm + RING, mirb = ctb + (unsigned)offsetof(Ctl, mir);
  const v4u frs = make_rsrc(myws + OFF_FLAG, (unsigned)(OFF_DBG - OFF_FLAG));
  const v4u drs = make_rsrc(myws + OFF_DATA + (size_t)(call & 1) * DATA_BYTES, DATA_BYTES);
  const unsigned ro = (unsigned)(OFF_RES - OFF_FLAG) + jb.agg * 128, vz = 0;
  auto stream = [&](int a) {
    return a >= NA ? -1 : a < NUN ? (jb.unit(a) >> 3 == me ? -1 : unit_stream(jb.unit(a))) : jb.wsrc == me ? -1 : jb.wsrc * NSLOT + 3 + a - NUN;
  };
  const int sf = stream(lane >> 4), sp = stream(EARLY ? lane / NSPEC : 0);
  const unsigned fo = sf * (NSTEP * 4) + (lane & 15) * 16, fo4 = stream(4) * (NSTEP * 4) + lane * 4;
  const unsigned po = (sp * NSTEP + (EARLY ? S0 + lane % NSPEC : lane)) * ITEM + (ITEM - 4);
#pragma unroll
  for (int a = 0; a < NA; ++a) ctl->mir[a][lane] = stream(a) < 0 ? call : 0u;
  ctl->prb[lane] = sp < 0;
  if (lane < 16) ((volatile LAS unsigned*)ctl)[lane] = 0u;
  if (EARLY) asm volatile("s_waitcnt lgkmcnt(0)" ::: "memory"); else BARRIER();
  __builtin_amdgcn_s_setprio(AGQ_PRIO);
  const unsigned long long t0 = __builtin_amdgcn_s_memrealtime();
  unsigned long long tl = t0 - AGQ_POLL, now = t0, all = 0ull, safe = 0ull;
  unsigned spin = 0, round = 0;
  bool bad = false;
  auto look = [&](bool res) {
    now = __builtin_amdgcn_s_memrealtime();
    bool fast = false, probe = false;
    if (safe != ~0ull) {
      asm volatile("" ::: "memory");
      const LAS unsigned* mp = (const LAS unsigned*)(sm + RING + offsetof(Ctl, mir)) + lane;
      unsigned m[NA];
#pragma unroll
      for (int a = 0; a < NA; ++a) m[a] = mp[a * NSTEP];
      const unsigned pr = mp[5 * NSTEP];
      const unsigned long long pb = __ballot(pr != 0u);
      const unsigned long long was = all;
      all = safe = ~0ull;
#pragma unroll
      for (int a = NA - 1; a >= 0; --a) {
        const unsigned long long b = __ballot(m[a] >= call);
        if (!EARLY && a == 0) fast = safe != ~0ull;
        safe &= b;
        all &= b | (EARLY ? pb >> (a * NSPEC) << S0 : a == 0 ? pb : 0ull);
        if (!EARLY && a == 0) probe = pr == 0u && m[0] < call && lane - __builtin_ctzll(~(b | pb) | 1ull << 63) < AGQ_WIN;
      }
      if ((all & ~was) >> lane & 1) TRACE(lane, 2);
      if (EARLY) { all |= all >> (S0 - AGQ_EK) << S0; fast = (~safe << NSPEC) != 0ull; probe = pr == 0u && (~all << (NSPEC + AGQ_EWIN)) == 0ull && all != ~0ull; }
    }
    if (now - tl >= AGQ_POLL && !bad) {
      tl = now;
      asm volatile("s_nop 4");
      if (probe) DMA4(mirb + 5 * (NSTEP * 4), po, drs);
      if (fast || (safe != ~0ull && (++round & 7) == 0)) {
        if (sf >= 0) DMA16(mirb, fo, frs, 0);
        if (EARLY && jb.wsrc != me) DMA4(mirb + 4 * (NSTEP * 4), fo4, frs);
      }
      if (res && lane == 0) DMA16(ctb, vz, frs, ro);
    }
  };
  auto idle = [&]() {
    if ((++spin & 255) == 0 && (__builtin_amdgcn_readfirstlane(__hip_atomic_load(tmo, AGT_RLX)) == call || now - t0 > DEADLINE_TICKS)) {
      if (lane == 0) __hip_atomic_store(tmo, call, AGT_RLX);
      asm volatile("s_waitcnt vmcnt(0)" ::: "memory");
#pragma unroll
      for (int a = 0; a < NA; ++a) ctl->mir[a][lane] = call;
      if (lane == 0) ctl->res = v2u{0u, ~(2 * call)};
      bad = true;
    }
    __builtin_amdgcn_s_sleep(1);
  };
  if (EARLY) {
    unsigned long long used = 0ull;
    for (int n = 0; n < NSTEP; ++n) {
      while (true) {
        look(n >= NSTEP - 2);
        if (all & ~used) break;
        idle();
      }
      const int st = __builtin_ctzll(all & ~used);
      used |= 1ull << st;
      if (lane == 0) ctl->sched[n] = st | (unsigned)(safe >> st & 1) << 6;
      if (n == NSTEP - 1) STAMP(1);
      BARRIER();
    }
#pragma unroll
    for (int n = 0; n < E_PF; ++n) { look(true); BARRIER(); }
  } else {
    do { look(__builtin_popcountll(all) >= NSTEP - 8); idle(); } while (all != ~0ull);
    STAMP(1);
  }
  while (!bad) {
    look(true);
    if (__builtin_amdgcn_readfirstlane(ctl->res.y) == 2 * call + 1) break;
    idle();
  }
  asm volatile("s_waitcnt vmcnt(0)" ::: "memory");
  STAMP(3);
}

template <int MT>
__device__ __forceinline__ void finish(const f32x4 (&acc)[MT][4], const Params& p, const Job jb, char* const myws, LAS char* const sm,
                                       char* tw, char* ob, char* dn, const int tm0, const int col,
                                       const unsigned call, const int lane, const int wave) {
  volatile LAS Ctl* const ctl = (volatile LAS Ctl*)(sm + RING);
  const bool foreign = jb.wsrc != p.rank;
  STAMP(1);
  float m = 0.f;
#pragma unroll
  for (int a = 0; a < MT; ++a)
#pragma unroll
    for (int b = 0; b < 4; ++b)
#pragma unroll
      for (int r = 0; r < 4; ++r) m = fmaxf(m, acc[a][b][r]);
  unsigned mu = wave_max_u32(__float_as_uint(m));
  unsigned tmx = 0, cnt = 0;
  asm volatile("s_waitcnt vmcnt(0)" ::: "memory");
  if (lane == 0) {
    __hip_atomic_fetch_max((LAS unsigned*)&ctl->tmax, mu, WG_RLX);
    cnt = __hip_atomic_fetch_add((LAS unsigned*)&ctl->nmax, 1u, WG_RLX);
    tmx = ctl->tmax;
  }
  mu = __builtin_amdgcn_readfirstlane(tmx);
  if (__builtin_amdgcn_readfirstlane(cnt) == 3) {
    if (lane < NR) __hip_atomic_store((GAS unsigned long long*)tw, ((unsigned long long)call << 32) | mu, SYS_RLX);
    if (lane == 0) __hip_atomic_store((GAS unsigned*)(myws + OFF_CTR) + blockIdx.x, call, AGT_RLX);
    STAMP(5);
  }
  const __amdgpu_buffer_rsrc_t ors = __builtin_amdgcn_make_buffer_rsrc((void*)ob, 0, NR * MSH * NSH, 0x00020000);
  const unsigned ooff = (unsigned)(lane & 15) * NSH + col + (lane >> 4) * 16;
  unsigned sent = 0xffffffffu, nsent = 0, state = 0;
  bool shut = foreign;
  const unsigned long long t0 = __builtin_amdgcn_s_memrealtime();
  auto look = [&]() {
    const v2u c = ctl->res;
    const unsigned tag = __builtin_amdgcn_readfirstlane(c.y);
    state = tag == 2 * call ? 1u : tag == 2 * call + 1 ? 2u : tag == ~(2 * call) ? 3u : 0u;
    if (state) mu = umax(mu, __builtin_amdgcn_readfirstlane(c.x));
    if (state == 1 && shut) {
      const unsigned gt = __builtin_amdgcn_readfirstlane(ctl->gate.x);
      shut = gt >> 4 != (call & 0xfffffffu) || !(gt >> (jb.wsrc & 3) & 1);
      if (shut) state = 0;
    }
  };
  while (true) {
    look();
    if (__builtin_amdgcn_s_memrealtime() - t0 > 2 * DEADLINE_TICKS) state = 3;
    if (state != 0 && (mu != sent || state == 3)) {
      if (nsent == 0) STAMP(2);
      const unsigned cur = mu;
      const bool bad = state == 3;
      const float scale = __uint_as_float(cur) / 127.0f;
      const float inv = scale > 0.f ? 1.0f / scale : 0.f;
      ++nsent;
#pragma unroll
      for (int mt = 0; mt < MT; ++mt) {
        const int tm = tm0 + mt;
        if (tm >= jb.nvalid) break;
        if (mt >= AGQ_INFL && state == 1 && foreign) {
          asm volatile("s_waitcnt vmcnt(%0)" :: "i"(AGQ_INFL - 1) : "memory");
          look();
          if (mu != cur || state == 3) break;
        }
        unsigned d[4];
#pragma unroll
        for (int nt = 0; nt < 4; ++nt) {
          unsigned v = 0;
#pragma unroll
          for (int r = 0; r < 4; ++r) {
            const float qf = fminf(__builtin_rintf(fmaxf(acc[mt][nt][r], 0.f) * inv), 127.0f);
            v |= (unsigned)(int)qf << (8 * r);
          }
          d[nt] = bad ? 0x80808080u : v;
        }
        { auto r = __builtin_amdgcn_permlane32_swap(d[0], d[2], false, false); d[0] = r[0]; d[2] = r[1]; }
        { auto r = __builtin_amdgcn_permlane32_swap(d[1], d[3], false, false); d[1] = r[0]; d[3] = r[1]; }
        { auto r = __builtin_amdgcn_permlane16_swap(d[0], d[1], false, false); d[0] = r[0]; d[1] = r[1]; }
        { auto r = __builtin_amdgcn_permlane16_swap(d[2], d[3], false, false); d[2] = r[0]; d[3] = r[1]; }
        const v4u o = {d[0], d[1], d[2], d[3]};
        __builtin_amdgcn_raw_buffer_store_b128(o, ors, ooff, (unit_row(jb.unit(tm >> 2)) + (tm & 3) * 16) * NSH, 17);
      }
      asm volatile("s_waitcnt vmcnt(0)" ::: "memory");
      if (bad) break;
      if (mu != cur || state == 3) continue;
      if (foreign && lane == 0) __hip_atomic_store((GAS unsigned long long*)dn, ((unsigned long long)call << 32) | cur, SYS_RLX);
      sent = cur;
      STAMP(3);
    }
    if (state == 2) break;
    __builtin_amdgcn_s_sleep(1);
  }
  STAMPV(4, nsent);
}

template <bool EARLY>
__device__ __forceinline__ void gemm_tile(const Params& p, const Job jb, char* const myws, LAS char* const sm,
                                          const unsigned call, const int lane, const int wave) {
  if (wave == 7) { poll_wave<EARLY>(p, jb, myws, sm, call, lane, wave); return; }
  const int me = p.rank;
  const bool foreign = jb.wsrc != me;
  volatile LAS Ctl* const ctl = (volatile LAS Ctl*)(sm + RING);
  const unsigned smb = (unsigned)(size_t)sm;
  const v4u drs = make_rsrc(myws + OFF_DATA + (size_t)(call & 1) * DATA_BYTES, DATA_BYTES);
  const v4u wrs = foreign ? drs : make_rsrc(p.w, KDIM * NSH * 2);
  const unsigned ks = foreign ? 256 : NSH * 2, wt = foreign ? ITEM : BK * NSH * 2, hs = foreign ? STREAM_BYTES : 32 * NSH * 2;
  const unsigned wb = (foreign ? (jb.wsrc * NSLOT + 3) * STREAM_BYTES : jb.wct * 256) + jb.half * 128;
  if (EARLY && wave >= 4) {
    const int l = wave - 4, ur = jb.unit(l);
    const bool loc = ur >> 3 == me;
    const unsigned rs = loc ? KDIM * 2 : 128;
    const v4u xrs = loc ? make_rsrc(p.x, MSH * KDIM * 2) : drs;
    const unsigned xb = loc ? (ur & 7) * 64 * (KDIM * 2) : unit_stream(ur) * STREAM_BYTES, xt = loc ? BK * 2 : ITEM;
    v16u vo;
#pragma unroll
    for (int i = 0; i < 8; ++i) vo[i] = (i * 8 + (lane >> 3)) * rs + (((lane & 7) ^ ((4 * i + (lane >> 4)) & 7)) << 4) - (i & 3) * 1024;
#pragma unroll
    for (int i = 0; i < 8; ++i) {
      const int k = 5 * l + i;
      vo[8 + i] = (k >> 3) * hs + ((k & 7) * 4 + (lane >> 4)) * ks + (((lane & 15) ^ (((lane >> 4) | ((k & 2) << 1)) << 1)) << 4) - (i & 3) * 1024;
    }
    auto issue = [&](const unsigned st, const unsigned lo) {
      asm volatile("s_mov_b32 m0, %[lx]\n\ts_nop 3\n\t"
                   PDMA(0, "rx", "sx", 0) PDMA(1, "rx", "sx", 1024) PDMA(2, "rx", "sx", 2048) PDMA(3, "rx", "sx", 3072) M0ADD
                   PDMA(4, "rx", "sx", 0) PDMA(5, "rx", "sx", 1024) PDMA(6, "rx", "sx", 2048) PDMA(7, "rx", "sx", 3072)
                   "s_mov_b32 m0, %[lw]\n\ts_nop 0\n\t"
                   PDMA(8, "rw", "sw", 0) PDMA(9, "rw", "sw", 1024) PDMA(10, "rw", "sw", 2048) PDMA(11, "rw", "sw", 3072) M0ADD
                   PDMA(12, "rw", "sw", 0) PDMA(13, "rw", "sw", 1024)
                   :: "{v[80:95]}"(vo), [lx] "s"(smb + lo + l * 8192), [lw] "s"(smb + lo + E_XB + l * 5120), [rx] "s"(xrs), [rw] "s"(wrs),
                      [sx] "s"(xb + st * xt), [sw] "s"(wb + st * wt)
                   : "memory", "scc");
    };
    const int fa = lane < 3 ? (jb.unit(lane) >> 3 == me ? -1 : unit_stream(jb.unit(lane))) : lane < 5 && foreign ? jb.wsrc * NSLOT + lane : -1;
    const unsigned long long t0 = __builtin_amdgcn_s_memrealtime();
#define CRD(i, a, off) "ds_read_b128 " VR(128, i) ", %[" a "] offset:" #off "\n\t"
#define CMN(i) MN(128, 4*i) MN(128, 4*i+2)
#define SETTLE(YOUNG, h, lo) \
    if ((h) & NSTEP) asm volatile("s_waitcnt vmcnt(%0)" :: "i"(YOUNG) : "memory"); \
    else while (true) { \
      unsigned mn; \
      f32x32 T0, T1; \
      asm volatile("v_mov_b32 %[mn], -1\n\t" \
                   "s_waitcnt vmcnt(%[y]+10)\n\t" CRD(0, "a", 0) CRD(1, "a", 1024) CRD(2, "a", 2048) CRD(3, "a", 3072) \
                   "s_waitcnt vmcnt(%[y]+6)\n\t" CRD(4, "a", 4096) CRD(5, "a", 5120) CRD(6, "a", 6144) CRD(7, "a", 7168) \
                   "s_waitcnt lgkmcnt(4)\n\t" CMN(0) CMN(1) CMN(2) CMN(3) \
                   "s_waitcnt vmcnt(%[y]+2)\n\t" CRD(8, "b", 0) CRD(9, "b", 1024) CRD(10, "b", 2048) CRD(11, "b", 3072) \
                   "s_waitcnt lgkmcnt(4)\n\t" CMN(4) CMN(5) CMN(6) CMN(7) \
                   "s_waitcnt vmcnt(%[y])\n\t" CRD(12, "b", 4096) CRD(13, "b", 5120) \
                   "s_waitcnt lgkmcnt(2)\n\t" CMN(8) CMN(9) CMN(10) CMN(11) \
                   "s_waitcnt lgkmcnt(0)\n\t" CMN(12) CMN(13) \
                   : [mn] "=&v"(mn), "=&{v[128:159]}"(T0), "=&{v[160:191]}"(T1) \
                   : [a] "v"(smb + (lo) + l * 8192 + lane * 16), [b] "v"(smb + (lo) + E_XB + l * 5120 + lane * 16), [y] "i"(YOUNG) : "memory"); \
      if (!__any(mn == 0u)) break; \
      bool ok = false; \
      if ((++nfail & 3) == 0) { \
        GAS unsigned* const tmo = (GAS unsigned*)(myws + OFF_TMO); \
        const unsigned f = fa < 0 ? call : __hip_atomic_load((GAS unsigned*)(myws + OFF_FLAG) + fa * NSTEP + (h), SYS_RLX); \
        const bool late = __builtin_amdgcn_readfirstlane(__hip_atomic_load(tmo, AGT_RLX)) == call || __builtin_amdgcn_s_memrealtime() - t0 > DEADLINE_TICKS; \
        if (late && lane == 0) __hip_atomic_store(tmo, call, AGT_RLX); \
        ok = late || __all(f >= call); \
      } \
      issue(h, lo); \
      asm volatile("s_waitcnt vmcnt(0)" ::: "memory"); \
      if (ok) break; \
    } \
    asm volatile("s_barrier" ::: "memory")
    static_assert(E_PF == 2 && E_NBUF == 3, "loading wave");
    unsigned lo = 0, h0 = NSTEP, h1 = NSTEP, nfail = 0;
    for (int n = 0; n < NSTEP; ++n) {
      const unsigned nx = lo + E_STAGE == E_NBUF * E_STAGE ? 0 : lo + E_STAGE;
      SETTLE(E_NLD, h0, nx);
      h0 = h1;
      h1 = __builtin_amdgcn_readfirstlane(ctl->sched[n]) & (2 * NSTEP - 1);
      issue(h1 & (NSTEP - 1), lo);
      lo = nx;
    }
    SETTLE(E_NLD, h0, (NSTEP - 2) % E_NBUF * E_STAGE);
    SETTLE(0, h1, (NSTEP - 1) % E_NBUF * E_STAGE);
    STAMPV(4, nfail);
    return;
  }
  char* tw = p.ws[0];
#pragma unroll
  for (int i = 1; i < NR; ++i) tw = (lane == i) ? p.ws[i] : tw;
  tw += OFF_AMAX + (size_t)(call & 1) * AMAX_BYTES + (size_t)jb.gran * 8;
  char* ob = pick_ptr(p.out, jb.wsrc);
  char* dn = pick_ptr(p.ws, jb.wsrc) + OFF_DONE + (jb.done + (wave & 3)) * 8;
  asm volatile("" : "+v"(tw), "+s"(ob), "+s"(dn));
  const int fr = lane & 15, fg = lane >> 4, q = (lane >> 2) & 3, pp = lane & 3;
  constexpr int MT = 6, WROW = EARLY ? 256 : 128;
#define READ_W(sb, wrd) \
    bf16x8 wfr[2][4]; \
    _Pragma("unroll") for (int kk = 0; kk < 2; ++kk) \
      _Pragma("unroll") for (int nt = 0; nt < 4; ++nt) { \
        LAS char* a = (sb) + (((wrd) ^ (nt << 5)) + kk * 32 * WROW); \
        const s16x4 lo = __builtin_amdgcn_ds_read_tr16_b64_v4i16((LAS s16x4*)a); \
        const s16x4 hi = __builtin_amdgcn_ds_read_tr16_b64_v4i16((LAS s16x4*)(a + 4 * WROW)); \
        wfr[kk][nt] = __builtin_bit_cast(bf16x8, __builtin_shufflevector(lo, hi, 0, 1, 2, 3, 4, 5, 6, 7)); \
      }
  if constexpr (EARLY) {
    const int wm = wave >> 1, wn = wave & 1;
    f32x4 acc[MT][4];
#pragma unroll
    for (int a = 0; a < MT; ++a)
#pragma unroll
      for (int b = 0; b < 4; ++b) acc[a][b] = f32x4{0.f, 0.f, 0.f, 0.f};
    const unsigned xrd = (wm * 96 + fr) * 128 + ((fg ^ (fr >> 1)) << 4);
    const unsigned wrd = E_XB + (fg * 8 + q) * WROW + ((((wn << 2) ^ (q | ((fg & 1) << 2))) << 5) | (pp << 3));
#pragma unroll
    for (int n = 0; n < E_PF; ++n) BARRIER();
    unsigned so = 0;
    for (int t = 0; t < NSTEP; ++t) {
      BARRIER();
      LAS char* sb = sm + so;
      READ_W(sb, wrd);
      bf16x8 xf[2][MT];
#pragma unroll
      for (int kk = 0; kk < 2; ++kk)
#pragma unroll
        for (int mt = 0; mt < MT; ++mt) xf[kk][mt] = *(const LAS bf16x8*)(sb + ((xrd ^ (kk << 6)) + mt * 2048));
      __builtin_amdgcn_sched_barrier(0);
#pragma unroll
      for (int kk = 0; kk < 2; ++kk)
#pragma unroll
        for (int mt = 0; mt < MT; ++mt)
#pragma unroll
          for (int nt = 0; nt < 4; ++nt)
            acc[mt][nt] = __builtin_amdgcn_mfma_f32_16x16x32_bf16(wfr[kk][nt], xf[kk][mt], acc[mt][nt], 0, 0, 0);
      so = so + E_STAGE == E_NBUF * E_STAGE ? 0 : so + E_STAGE;
    }
    finish<MT>(acc, p, jb, myws, sm, tw, ob, dn, wm * MT, jb.wct * 128 + wn * 64, call, lane, wave);
  } else {
    const bool loc = jb.ur0 >> 3 == me;
    const v4u xrs = loc ? make_rsrc(p.x, MSH * KDIM * 2) : drs;
    const unsigned xb = loc ? (jb.ur0 & 7) * 64 * (KDIM * 2) : unit_stream(jb.ur0) * STREAM_BYTES, xt = loc ? BK * 2 : ITEM;
    v16u vo;
#pragma unroll
    for (int i = 0; i < 8; ++i) {
      vo[i] = (i >> 2) * hs + ((i & 3) * 8 + (lane >> 3)) * ks + (((lane & 7) ^ ((fg & 1) << 1) ^ ((i & 1) << 2)) << 4) - (i & 3) * 1024;
      vo[8 + i] = loc ? ((i >> 1) * 16 + fr) * (KDIM * 2) + (i & 1) * 64 + fg * 16 : i * 1024 + lane * 16;
    }
    const unsigned wrd = (fg * 8 + q) * WROW + ((((q >> 1) | ((fg & 1) << 1)) << 5) | (pp << 3));
    LAS char* const my = sm + work_lds(wave);
    const unsigned wa = (unsigned)(size_t)my + wrd;
    f32x32 XA = 0.f, XB = 0.f, C0 = 0.f, C1 = 0.f, T = 0.f;
    f32x32 R0 = 0.f, R1 = 0.f;
    const LAS unsigned* const mp = (const LAS unsigned*)(sm + RING + offsetof(Ctl, mir)) + lane;
    BARRIER();
    const unsigned long long t0 = __builtin_amdgcn_s_memrealtime();
#define XLD(b, i) "buffer_load_dwordx4 " VR(b, i) ", v[88+" #i "], %[rs], %[so] offen sc0 sc1\n\t"
#define WD(i, off) PDMA(i, "wrs", "wo", off)
#define XLD8(b) XLD(b, 0) XLD(b, 1) XLD(b, 2) XLD(b, 3) XLD(b, 4) XLD(b, 5) XLD(b, 6) XLD(b, 7)
#define MFL(b, kk, nt, A, B) MF(b, kk, 0, nt) A MF(b, kk, 1, nt) MF(b, kk, 2, nt) B MF(b, kk, 3, nt)
#define WRD(r, kk, nt) "ds_read_b64_tr_b16 v[" #r "+16*" #kk "+4*" #nt ":" #r "+16*" #kk "+4*" #nt "+1], %[a" #nt "] offset:4096*" #kk "\n\t" \
                       "ds_read_b64_tr_b16 v[" #r "+16*" #kk "+4*" #nt "+2:" #r "+16*" #kk "+4*" #nt "+3], %[a" #nt "] offset:4096*" #kk "+512\n\t"
#define WRD8(r) WRD(r, 0, 0) WRD(r, 0, 1) WRD(r, 0, 2) WRD(r, 0, 3) WRD(r, 1, 0) WRD(r, 1, 1) WRD(r, 1, 2) WRD(r, 1, 3)
#define MV(r, i) "v_mov_b64 v[96+" #i ":97+" #i "], v[" #r "+" #i ":" #r "+1+" #i "]\n\t"
#define MV4(r, i) MV(r, i) MV(r, i+2) MV(r, i+4) MV(r, i+6)
#define MV16(r) MV4(r, 0) MV4(r, 8) MV4(r, 16) MV4(r, 24)
#define MF(b, kk, mt, nt) "v_mfma_f32_16x16x32_bf16 " VR(192, 4*mt+nt) ", " VR(96, 4*kk+nt) ", " VR(b, 2*mt+kk) ", " VR(192, 4*mt+nt) "\n\t"
#define MF4(b, kk, nt) MF(b, kk, 0, nt) MF(b, kk, 1, nt) MF(b, kk, 2, nt) MF(b, kk, 3, nt)
#define CHECK(x, L) "s_cmp_lt_i32 %[cs], 3\n\ts_cbranch_scc1 Lr" L "%=\n\ts_cmp_eq_u32 %[cs], 3\n\ts_cbranch_scc1 Ls" L "%=\n\t" \
                 MV16(48) "s_branch Lt" L "%=\nLs" L "%=:\n\t" MV16(16) "s_branch Lt" L "%=\nLr" L "%=:\n\t" WRD8(96) "Lt" L "%=:\n\t" \
                 "v_min_u32 %[mn], v[" #x "], v[" #x "+1]\n\t" MN(x, 2) MN(x, 4) MN(x, 6) MN(x, 8) MN(x, 10) MN(x, 12) MN(x, 14) MN(x, 16) \
                 MN(x, 18) MN(x, 20) MN(x, 22) MN(x, 24) MN(x, 26) MN(x, 28) MN(x, 30) "v_cmp_gt_u32 vcc, %[full], %[mn]\n\t"
#define PHASE(x, y) { \
      int ws, st; \
      unsigned long long sf; \
      while (true) { \
        asm volatile("" ::: "memory"); \
        const unsigned a0 = mp[0], a1 = mp[NSTEP], a2 = mp[2 * NSTEP], a3 = mp[5 * NSTEP]; \
        sf = __ballot(a0 >= call); \
        const unsigned long long sn = sf | __ballot(a3 != 0u); \
        ws = wnext < NSTEP && (fre & 7) && (__ballot(a1 >= call && a2 >= call) >> wnext & 1) ? __builtin_ctz(fre) : -1; \
        const unsigned long long av = (sn | sn >> (NSTEP - LAG) << (NSTEP - LAG + AGQ_LK)) & (ws < 0 ? have : have | 1ull << wnext); \
        st = av ? __builtin_ctzll(av) : -1; \
        if ((ws & st & full) >= 0 || (left = left && (unsigned)__builtin_amdgcn_s_memrealtime() - (unsigned)t0 <= 2 * (unsigned)DEADLINE_TICKS ? left : 0) == 0) break; \
        if (AGQ_WREG && fre >= 8 && (~fre & lds) && wnext < NSTEP) { \
            \
          const int b_ = __builtin_ctz(~fre & lds), r_ = __builtin_ctz(fre & 24); \
          const unsigned pa = wa + b_ * ITEM; \
          asm volatile("s_waitcnt vmcnt(0)\n\ts_cmp_eq_u32 %[r], 3\n\ts_cbranch_scc1 Lp%=\n\t" WRD8(48) "s_branch Lq%=\nLp%=:\n\t" WRD8(16) \
                       "Lq%=:\n\ts_waitcnt lgkmcnt(0)" \
                       : "+{v[16:47]}"(R0), "+{v[48:79]}"(R1) \
                       : [r] "s"(r_), [a0] "v"(pa), [a1] "v"(pa ^ 32), [a2] "v"(pa ^ 64), [a3] "v"(pa ^ 96) : "memory", "scc"); \
          slot = (slot & ~(255ull << 8 * r_)) | (slot >> 8 * b_ & 255ull) << 8 * r_ | 255ull << 8 * b_; \
          fre ^= 1 << b_ | 1 << r_; \
        } else __builtin_amdgcn_s_sleep(1); \
      } \
      if (left == 0) break; \
      const unsigned so_ = xb + st * xt, wo_ = wb + wnext * wt; \
      unsigned mn_; \
      if (ws >= 0) { have |= 1ull << wnext; slot = (slot & ~(255ull << 8 * ws)) | (unsigned long long)wnext << 8 * ws; fre &= ~(1 << ws); wnext += NWORK; } \
      if (st >= 0) { have &= ~(1ull << st); TRACE(st, 0); } \
      const unsigned ca = wa + cs * ITEM; \
      asm volatile("s_mov_b32 m0, %[lb]\n\tv_mov_b32 v[" #y "+31], 0\n\t" \
                   "s_cmp_lt_i32 %[full], 0\n\ts_cbranch_scc1 Lfirst%=\n\t" \
                   "v_cmp_ne_u32 vcc, 0, v[" #x "+31]\n\ts_cbranch_vccz Lfirst%=\n\t" \
                     \
                   "s_waitcnt vmcnt(0)\n\t" CHECK(x, "a") "s_cbranch_vccnz Lbad%=\n\t" \
                   "s_cmp_lt_i32 %[ws], 0\n\ts_cbranch_scc1 Lnw%=\n\t" \
                   "s_waitcnt lgkmcnt(14)\n\t" MFL(x, 0, 0, WD(0, 0), WD(1, 1024)) "s_waitcnt lgkmcnt(12)\n\t" MFL(x, 0, 1, WD(2, 2048), WD(3, 3072)) \
                   "s_add_u32 m0, m0, 4096\n\t" \
                   "s_waitcnt lgkmcnt(10)\n\t" MFL(x, 0, 2, WD(4, 0), WD(5, 1024)) "s_waitcnt lgkmcnt(8)\n\t" MFL(x, 0, 3, WD(6, 2048), WD(7, 3072)) \
                   "s_cmp_lt_i32 %[st], 0\n\ts_cbranch_scc1 Lhalf%=\n\t" \
                   "s_waitcnt lgkmcnt(6)\n\t" MFL(x, 1, 0, XLD(y, 0), XLD(y, 1)) "s_waitcnt lgkmcnt(4)\n\t" MFL(x, 1, 1, XLD(y, 2), XLD(y, 3)) \
                   "s_waitcnt lgkmcnt(2)\n\t" MFL(x, 1, 2, XLD(y, 4), XLD(y, 5)) "s_waitcnt lgkmcnt(0)\n\t" MFL(x, 1, 3, XLD(y, 6), XLD(y, 7)) \
                   "s_branch Lskip%=\n" \
                   "Lnw%=:\n\ts_cmp_lt_i32 %[st], 0\n\ts_cbranch_scc1 Lgo%=\n\t" \
                   "s_waitcnt lgkmcnt(14)\n\t" MFL(x, 0, 0, XLD(y, 0), XLD(y, 1)) "s_waitcnt lgkmcnt(12)\n\t" MFL(x, 0, 1, XLD(y, 2), XLD(y, 3)) \
                   "s_waitcnt lgkmcnt(10)\n\t" MFL(x, 0, 2, XLD(y, 4), XLD(y, 5)) "s_waitcnt lgkmcnt(8)\n\t" MFL(x, 0, 3, XLD(y, 6), XLD(y, 7)) \
                   "s_branch Lhalf%=\n" \
                   "Lbad%=:\n\ts_mov_b32 %[full], -2\n\ts_waitcnt lgkmcnt(0)\n" \
                     \
                   "Lfirst%=:\n\ts_cmp_lt_i32 %[ws], 0\n\ts_cbranch_scc1 Lx%=\n\t" \
                   WD(0, 0) WD(1, 1024) WD(2, 2048) WD(3, 3072) M0ADD WD(4, 0) WD(5, 1024) WD(6, 2048) WD(7, 3072) \
                   "s_cmp_lt_i32 %[st], 0\n\ts_cbranch_scc1 Lone%=\n\t" XLD8(y) \
                   "s_cmp_lt_i32 %[full], 0\n\ts_cbranch_scc1 Lskip%=\n\ts_waitcnt vmcnt(16)\n\ts_branch Ldone%=\n" \
                   "Lx%=:\n\ts_cmp_lt_i32 %[st], 0\n\ts_cbranch_scc1 Lnone%=\n\t" XLD8(y) \
                   "Lone%=:\n\ts_cmp_lt_i32 %[full], 0\n\ts_cbranch_scc1 Lskip%=\n\ts_waitcnt vmcnt(8)\n\ts_branch Ldone%=\n" \
                   "Lnone%=:\n\ts_cmp_lt_i32 %[full], 0\n\ts_cbranch_scc1 Lskip%=\n\ts_waitcnt vmcnt(0)\n" \
                   "Ldone%=:\n\t" CHECK(x, "b") "s_cbranch_vccz Lgo%=\n\t" \
                   "s_mov_b32 %[full], -2\n\ts_waitcnt lgkmcnt(0)\n\ts_branch Lskip%=\n" \
                   "Lgo%=:\n\t" \
                   "s_waitcnt lgkmcnt(14)\n\t" MF4(x, 0, 0) "s_waitcnt lgkmcnt(12)\n\t" MF4(x, 0, 1) \
                   "s_waitcnt lgkmcnt(10)\n\t" MF4(x, 0, 2) "s_waitcnt lgkmcnt(8)\n\t" MF4(x, 0, 3) \
                   "Lhalf%=:\n\t" \
                   "s_waitcnt lgkmcnt(6)\n\t" MF4(x, 1, 0) "s_waitcnt lgkmcnt(4)\n\t" MF4(x, 1, 1) \
                   "s_waitcnt lgkmcnt(2)\n\t" MF4(x, 1, 2) "s_waitcnt lgkmcnt(0)\n\t" MF4(x, 1, 3) \
                   "Lskip%=:" \
                   : "+{v[128:159]}"(XA), "+{v[160:191]}"(XB), "+{v[192:223]}"(C0), "+{v[224:255]}"(C1), "+{v[96:127]}"(T), \
                     [full] "+s"(full), [mn] "=&v"(mn_) \
                   : "{v[80:95]}"(vo), "{v[16:47]}"(R0), "{v[48:79]}"(R1), [cs] "s"(cs), [rs] "s"(xrs), [so] "s"(so_), [wo] "s"(wo_), [st] "s"(st), [ws] "s"(ws), [lb] "s"(smb + work_lds(wave) + ws * ITEM), \
                     [wrs] "s"(wrs), [a0] "v"(ca), [a1] "v"(ca ^ 32), [a2] "v"(ca ^ 64), [a3] "v"(ca ^ 96) \
                   : "memory", "scc", "vcc"); \
      full = __builtin_amdgcn_readfirstlane(full); \
      if (full == -2) { have |= 1ull << cur; ++nfail; TRACE(cur, 3); } \
      else if (full >= 0) { fre |= 1 << cs; TRACE(cur, 1); if (--left == 0) break; } \
      full = st < 0 ? -1 : (int)(~sf >> st & 1); \
      cur = st; \
      cs = ((unsigned)slot & 255u) == (unsigned)st ? 0 : ((unsigned)slot >> 8 & 255u) == (unsigned)st ? 1 : ((unsigned)slot >> 16 & 255u) == (unsigned)st ? 2 : \
           (unsigned)slot >> 24 == (unsigned)st ? 3 : 4; \
    }
    const int lds = wave < 5 ? 7 : 3;
    int full = -1, cur = 0, cs = 0, wnext = wave, fre = lds | 24, left = (NSTEP - 1 - wave) / NWORK + 1, nfail = 0;
    unsigned long long slot = ~0ull;
    unsigned long long have = 0;
    while (true) {
      PHASE(128, 160);
      PHASE(160, 128);
    }
    asm volatile("s_nop 7\n\ts_nop 7\n\ts_waitcnt vmcnt(0)"
                 : "+{v[192:223]}"(C0), "+{v[224:255]}"(C1), "+{v[128:159]}"(XA), "+{v[160:191]}"(XB) :: "memory");
#pragma unroll
    for (int i = 0; i < 8; ++i) {
      *(LAS f32x4*)(my + i * 1024 + lane * 16) = f32x4{C0[4 * i], C0[4 * i + 1], C0[4 * i + 2], C0[4 * i + 3]};
      *(LAS f32x4*)(my + i * 1024 + 8192 + lane * 16) = f32x4{C1[4 * i], C1[4 * i + 1], C1[4 * i + 2], C1[4 * i + 3]};
    }
    asm volatile("" ::: "memory");
    ctl->fin[wave] = 1u;
    if (wave >= 4) { STAMP(2); STAMPV(4, nfail); return; }
    f32x4 sum[1][4] = {};
    for (unsigned todo = (1u << NWORK) - 1, spin = 0; todo;) {
      unsigned rdy = (unsigned)__ballot(ctl->fin[lane & 7] != 0u) & todo;
      todo &= ~rdy;
      if ((++spin & 255) == 0 && __builtin_amdgcn_s_memrealtime() - t0 > 3 * DEADLINE_TICKS) todo = 0;
      for (; rdy; rdy &= rdy - 1) {
        asm volatile("" ::: "memory");
        const LAS char* src = sm + work_lds(__builtin_ctz(rdy)) + wave * 4096 + lane * 16;
#pragma unroll
        for (int nt = 0; nt < 4; ++nt) sum[0][nt] += *(const LAS f32x4*)(src + nt * 1024);
      }
    }
    finish<1>(sum, p, jb, myws, sm, tw, ob, dn, wave, jb.wct * 128 + jb.half * 64, call, lane, wave);
  }
}

__global__ void __launch_bounds__(NTHR, 1) ag_gemm_q8(Params p) {
  __shared__ __attribute__((aligned(16))) char smem[RING + sizeof(Ctl)];
  const int lane = threadIdx.x & 63;
  const int wave = __builtin_amdgcn_readfirstlane(threadIdx.x >> 6);
  const int bid = blockIdx.x;
  const char *px = p.x, *pw = p.w;
#define AGQ_XCC __builtin_amdgcn_s_getreg(20 | (3 << 11))
  int xcc = AGQ_XCC & 7;
  char *myws = p.myws, *dws = p.pd[xcc];
  int me = p.rank;
  asm volatile("" : "+s"(px), "+s"(pw), "+s"(myws), "+s"(dws), "+s"(me));
  const int g = me >> 2, j = me & 3;
  GAS unsigned* const tmo = (GAS unsigned*)(myws + OFF_TMO);
  GAS unsigned* const ctr = (GAS unsigned*)(myws + OFF_CTR) + bid;

  if (__builtin_expect(bid < NPUSH, 1)) {
    STAMP(0);
    GAS unsigned* const wctr = ctr + wave * 256;
    const unsigned vzero = 0, vone = 1;
    unsigned pos = (bid >> 3) * WW + wave;
    if (wave >= WW) return;
    for (unsigned add = 1;; add = 0) {
      const int dst = PUSH_DST1(me, xcc), v = pos % PV;
      unsigned got, call0;
      if (dst == me) {
        call0 = __hip_atomic_load(wctr, AGT_RLX);
        got = __hip_atomic_fetch_add((GAS unsigned*)(myws + OFF_TICK) + dst * 32 + 16 + v, lane == 0 ? add : 0u, AGT_RLX);
        if (!add || __builtin_amdgcn_readfirstlane(got - call0) == 0) {
          if (lane == 0) __hip_atomic_store(wctr, call0 + 1, AGT_RLX);
          return;
        }
      } else {
      unsigned cl;
      asm volatile("global_load_dword %0, %1, %2 sc1" : "=&v"(cl) : "v"(vzero), "s"(wctr) : "memory");
      const bool same = (dst >> 2) == g;
      const unsigned xlane = (unsigned)((same ? 0 : 320) + (lane >> 3)) * (KDIM * 2) + (lane & 7) * 16;
      const unsigned flane = (unsigned)((same ? 0 : 320) + (lane & 15)) * (KDIM * 2) + (lane >> 4) * 16;
      const unsigned wlane = (unsigned)(lane >> 4) * (NSH * 2) + (dst & 3) * 256 + (lane & 15) * 16;
      const char* flag = dws + OFF_FLAG + me * NSLOT * NSTEP * 4;
      const char* tick = myws + OFF_TICK + dst * 128;
      v4u A[8], B[8];
      auto POS = [same](int n) {
        const int m = n - 4 * LAG;
        const int e = m < 0 ? n & 3 : n >= NITEM - LAG ? 4 : m % NSLOT;
        const int t = m < 0 ? n >> 2 : n >= NITEM - LAG ? n - 4 * NSTEP : m / NSLOT + (e < 4 ? LAG : 0);
        return (same ? e : e < 4 ? e + 1 : 0) * NSTEP + t;
      };
#define PSRC(n) const int ps_ = POS(n), t_ = ps_ % NSTEP, s_ = ps_ / NSTEP; const bool w_ = !same && s_ >= 3, f_ = s_ == (same ? 4 : 0); \
        const v4u rs_ = make_rsrc(w_ ? pw : px, MSH * KDIM * 2); \
        const unsigned vo_ = w_ ? wlane : f_ ? flane : xlane; \
        unsigned so_ = w_ ? (t_ * BK + (s_ - 3) * 32) * (NSH * 2) : s_ * 64 * (KDIM * 2) + t_ * (BK * 2); \
        const unsigned i0_ = w_ ? 4 * NSH * 2 : f_ ? 64 : 8 * KDIM * 2, i1_ = w_ ? 4 * NSH * 2 : f_ ? 16 * KDIM * 2 - 64 : 8 * KDIM * 2
#define ST8 "buffer_store_dwordx4 %0, %[v0], %[drs], %[dso] offen sc0 sc1\n\t" \
            "buffer_store_dwordx4 %1, %[v0], %[drs], %[dso] offen offset:1024 sc0 sc1\n\t" \
            "buffer_store_dwordx4 %2, %[v0], %[drs], %[dso] offen offset:2048 sc0 sc1\n\t" \
            "buffer_store_dwordx4 %3, %[v0], %[drs], %[dso] offen offset:3072 sc0 sc1\n\t" \
            "buffer_store_dwordx4 %4, %[v1], %[drs], %[dso] offen sc0 sc1\n\t" \
            "buffer_store_dwordx4 %5, %[v1], %[drs], %[dso] offen offset:1024 sc0 sc1\n\t" \
            "buffer_store_dwordx4 %6, %[v1], %[drs], %[dso] offen offset:2048 sc0 sc1\n\t" \
            "buffer_store_dwordx4 %7, %[v1], %[drs], %[dso] offen offset:3072 sc0 sc1\n\t"
#define PLOAD(R, n) do { PSRC(n); \
        asm volatile("s_nop 4\n\t" LD8("") : "=&v"(R[0]), "=&v"(R[1]), "=&v"(R[2]), "=&v"(R[3]), "=&v"(R[4]), "=&v"(R[5]), "=&v"(R[6]), "=&v"(R[7]), \
                     [so] "+s"(so_) : [vo] "v"(vo_), [rs] "s"(rs_), [i0] "s"(i0_), [i1] "s"(i1_) : "memory", "scc"); } while (0)
      PLOAD(A, v);
      PLOAD(B, v + PV);
      v4u drs = make_rsrc(dws + OFF_DATA + (size_t)me * NSLOT * STREAM_BYTES, DATA_BYTES + NSLOT * STREAM_BYTES);
      asm volatile("" : "+s"(drs), "+s"(flag), "+s"(tick));
      asm volatile("s_waitcnt vmcnt(8)" : "+v"(cl) :: "memory");
      call0 = cl;
      const unsigned call = call0 + 1;
      const unsigned vst0 = lane * 16 + (call & 1) * DATA_BYTES, vst1 = vst0 + 4096;
      const unsigned tbase = (call - 1) * NDRAW - 4 * PV;
      int prev = -1, na = v, nb = v + PV, nc = v + 2 * PV;
#define PROUND(R, cur, vz_, one_) do { \
        const int ld_ = nc < NITEM ? nc : cur; \
        PSRC(ld_); \
        const unsigned dso_ = POS(cur) * ITEM, fo_ = POS(prev >= 0 ? prev : 0) * 4; \
        unsigned tk_; \
        asm volatile("s_cmp_lt_i32 %[nc], %[nitem]\n\ts_cselect_b64 exec, 1, 0\n\ts_nop 2\n\t" \
                     "global_atomic_add %[tk], %[vz], %[one], %[tick] sc0\n\t" \
                     "s_mov_b64 exec, -1\n\t" \
                     ST8 LD8("") \
                     "s_waitcnt vmcnt(16)\n\t" \
                     "s_cmp_ge_i32 %[prev], 0\n\ts_cselect_b64 exec, 1, 0\n\t" \
                     "global_store_dword %[fo], %[call], %[flag] sc0 sc1\n\t" \
                     "s_mov_b64 exec, -1" \
                     : "+v"(R[0]), "+v"(R[1]), "+v"(R[2]), "+v"(R[3]), "+v"(R[4]), "+v"(R[5]), "+v"(R[6]), "+v"(R[7]), \
                       [so] "+s"(so_), [tk] "=&v"(tk_) \
                     : [vo] "v"(vo_), [rs] "s"(rs_), [i0] "s"(i0_), [i1] "s"(i1_), [v0] "v"(vst0), [v1] "v"(vst1), [drs] "s"(drs), [dso] "s"(dso_), \
                       [nc] "s"(nc), [nitem] "i"(NITEM), [prev] "s"(prev), [vz] "v"(vz_), [one] "v"(one_), [tick] "s"(tick), [fo] "v"(fo_), \
                       [call] "v"(call), [flag] "s"(flag) \
                     : "memory", "scc"); \
        prev = cur; cur = nc; \
        got = tk_; \
        const unsigned u_ = __builtin_amdgcn_readfirstlane(tk_ - tbase); \
        nc = (nc < NITEM && u_ < (unsigned)NITEM) ? (int)u_ : NITEM; \
      } while (0)
      PROUND(A, na, 64u + v * 4, add);
      nc = v + 3 * PV;
      const bool mine = !add || __builtin_amdgcn_readfirstlane(got - call0) == 0;
      while (mine) {
        PROUND(B, nb, vzero, vone);
        if (na >= NITEM) break;
        PROUND(A, na, vzero, vone);
        if (nb >= NITEM) break;
      }
      asm volatile("s_waitcnt vmcnt(0)"
                   : "+v"(A[0]), "+v"(A[1]), "+v"(A[2]), "+v"(A[3]), "+v"(A[4]), "+v"(A[5]), "+v"(A[6]), "+v"(A[7]),
                     "+v"(B[0]), "+v"(B[1]), "+v"(B[2]), "+v"(B[3]), "+v"(B[4]), "+v"(B[5]), "+v"(B[6]), "+v"(B[7]) :: "memory");
      if (lane == 0) __hip_atomic_store((GAS unsigned*)flag + POS(prev), call, SYS_RLX);
      if (mine && lane == 0) __hip_atomic_store(wctr, call, AGT_RLX);
      STAMP(1);
      if (mine) return;
      }
      if (lane == 0) __hip_atomic_fetch_sub((GAS unsigned*)(myws + OFF_TICK) + dst * 32 + 16 + v, 1u, AGT_RLX);
      const unsigned long long t0 = __builtin_amdgcn_s_memrealtime();
      for (unsigned k = 0;; ++k) {
        xcc = k & 7; pos = (k >> 3) % PV;
        unsigned e = call0;
        bool ok = lane == 0 && __hip_atomic_compare_exchange_strong((GAS unsigned*)(myws + OFF_TICK) + PUSH_DST1(me, xcc) * 32 + 16 + pos, &e, call0 + 1,
                                                                    __ATOMIC_RELAXED, AGT_RLX);
        if (__any(ok)) break;
        if (__builtin_amdgcn_s_memrealtime() - t0 > DEADLINE_TICKS) {
          if (lane == 0) { __hip_atomic_store(tmo, call0 + 1, AGT_RLX); __hip_atomic_store(wctr, call0 + 1, AGT_RLX); }
          return;
        }
      }
      dws = p.pd[xcc];
    }
  }

  STAMP(0);
  const unsigned call = __builtin_amdgcn_readfirstlane(__hip_atomic_load(ctr, AGT_RLX)) + 1;
  const int gi = bid - NPUSH;
  if (gi >= NGEMM) {
    const unsigned long long t0 = __builtin_amdgcn_s_memrealtime();
    const __amdgpu_buffer_rsrc_t crs = __builtin_amdgcn_make_buffer_rsrc((void*)(myws + OFF_DATA + (size_t)(~call & 1) * DATA_BYTES), 0, DATA_BYTES, 0x00020000);
    unsigned wt = AGQ_CLRT;
    {
      const v4u xr = make_rsrc(px, MSH * KDIM * 2), wr = make_rsrc(pw, MSH * KDIM * 2);
      unsigned d0, d1;
      while (__builtin_amdgcn_s_memrealtime() - t0 < AGQ_PREF) __builtin_amdgcn_s_sleep(2);
      for (unsigned o = ((gi - NGEMM) * 8 + wave) * 8192 + lane * 128; o < MSH * KDIM * 2; o += 8 * NSWEEP * 8192)
        asm volatile("buffer_load_dword %0, %2, %3, 0 offen\n\tbuffer_load_dword %1, %2, %4, 0 offen\n\ts_waitcnt vmcnt(0)"
                     : "=&v"(d0), "=&v"(d1) : "v"(o), "s"(xr), "s"(wr) : "memory");
    }
    for (int i = (gi - NGEMM) * 8 + wave; i < 7 * NITEM * (8 / AGQ_CLRN); i += 8 * NSWEEP, wt += AGQ_CLRD) {
      while (__builtin_amdgcn_s_memrealtime() - t0 < wt) __builtin_amdgcn_s_sleep(2);
      const int src = (me + 1 + i % 7) & 7;
#pragma unroll
      for (int m = 0; m < AGQ_CLRN; ++m)
        __builtin_amdgcn_raw_buffer_store_b128(v4u{0u, 0u, 0u, 0u}, crs, lane * 16 + m * 1024, src * (NITEM * ITEM) + i / 7 * (AGQ_CLRN * 1024), 17);
    }
    asm volatile("s_waitcnt vmcnt(0)" ::: "memory");
    __syncthreads();
    if (threadIdx.x == 0) __hip_atomic_store(ctr, call, AGT_RLX);
    STAMP(1);
    return;
  }
  const int idx = gi >> 3, rb = AGQ_GMAP == 2 ? (idx < 8 ? idx >> 1 : (idx - 8) >> 2) : AGQ_GMAP ? gi & 3 : (gi & 7) >> 1;
  bool early = idx < 8;
  const int sel = idx & 1;
  const int half = early ? 0 : (idx >> 1) & 1;
  const int c4 = early ? idx >> 1 : (idx - 8) >> 2;
  const int ct = AGQ_GMAP == 2 ? gi & 7 : AGQ_GMAP ? c4 + (gi & 4) : c4 * 2 + (gi & 1);
  const int sa = g * 4 + rb, sb = (1 - g) * 4 + rb;
  const bool diag = sel == 1 && ct == 4 + rb;
  Job jb;
  jb.ur0 = early ? sa * 8 + 3 * sel : sel ? sb * 8 + 5 : sa * 8 + 4;
  jb.ur1 = !sel ? sa * 8 + 1 : diag ? sa * 8 + 3 : sb * 8 + 6;
  jb.ur2 = !sel ? sa * 8 + 2 : diag ? sa * 8 + 3 : sb * 8 + 7;
  jb.wsrc = ct >= 4 ? (1 - g) * 4 + (ct - 4) : me;
  jb.wct = ct >= 4 ? j : ct;
  jb.half = half;
  jb.nvalid = diag ? 4 : 12;
  jb.gran = early ? me * NEARLY + gi : NR * NEARLY + me * NLATE + gi - NEARLY;
  jb.done = j * 128 + (early ? (rb * 2 + sel) * 4 : 32 + ((rb * 2 + sel) * 2 + half) * 4);
  jb.agg = rb;

  if (!early && diag && half == 1) {
    volatile LAS v2u* const part = (volatile LAS v2u*)((LAS char*)smem + RING);
    if (wave < 6 && lane == 0) part[wave] = v2u{0u, 0u};
    asm volatile("s_waitcnt vmcnt(0)" ::: "memory");
    __syncthreads();
    const size_t aoff = OFF_AMAX + (size_t)(call & 1) * AMAX_BYTES;
    const unsigned long long t0 = __builtin_amdgcn_s_memrealtime();
    if (wave < 6) {
      const char* gp = myws + aoff + wave * 2048 + lane * 16;
      unsigned mu = 0, cnt = 0, last = 0;
      while (true) {
        unsigned tm = 0;
        if (cnt < 4) {
          v4u a, b;
          asm volatile("global_load_dwordx4 %0, %3, off sc0 sc1\n\t"
                       "global_load_dwordx4 %1, %3, off offset:1024 sc0 sc1\n\t"
                       "global_load_dword %2, %4, off sc1\n\t"
                       "s_waitcnt vmcnt(0)"
                       : "=&v"(a), "=&v"(b), "=&v"(tm) : "v"(gp), "v"(tmo) : "memory");
          cnt = (a.y == call) + (a.w == call) + (b.y == call) + (b.w == call);
          mu = umax(umax(a.y == call ? a.x : 0u, a.w == call ? a.z : 0u), umax(b.y == call ? b.x : 0u, b.w == call ? b.z : 0u));
        }
        const unsigned tot = __builtin_popcountll(__ballot(cnt & 1)) + 2 * __builtin_popcountll(__ballot(cnt & 2)) +
                             4 * __builtin_popcountll(__ballot(cnt & 4));
        if (tot != last) {
          const unsigned m = wave_max_u32(mu);
          if (lane == 0) part[wave] = v2u{m, tot};
          last = tot;
        }
        if (tot == 256 || __any(tm == call) || __builtin_amdgcn_s_memrealtime() - t0 > DEADLINE_TICKS) break;
        __builtin_amdgcn_s_sleep(1);
      }
      return;
    }
    if (wave == 6) {
      GAS unsigned* const gp = (GAS unsigned*)(myws + OFF_RES + rb * 128 + 8);
      for (unsigned open = 0, it = 1; open != 15; ++it) {
        __builtin_amdgcn_s_sleep(4);
        if (__builtin_amdgcn_readfirstlane(part[0].y + part[1].y) >= AGQ_GT) {
          const unsigned t = __hip_atomic_load((GAS unsigned*)(myws + OFF_TICK) + ((1 - g) * 4 + (lane & 3)) * 32, AGT_RLX);
          const unsigned m = (unsigned)__ballot(t - (call - 1) * NDRAW >= NDRAW - AGQ_GATE) & 15u;
          if (m != open && lane == 0) __hip_atomic_store(gp, call << 4 | m, AGT_RLX);
          open = m;
        }
        if ((it & 63) == 0 && (__builtin_amdgcn_readfirstlane(__hip_atomic_load(tmo, AGT_RLX)) == call ||
                               __builtin_amdgcn_s_memrealtime() - t0 > DEADLINE_TICKS)) break;
      }
      return;
    }
    char* tw = p.ws[0];
#pragma unroll
    for (int i = 1; i < NR; ++i) tw = (lane == i) ? p.ws[i] : tw;
    GAS unsigned long long* rp = (GAS unsigned long long*)(myws + OFF_RES + rb * 128);
    unsigned mx = 0, pub = 0;
    bool bad = false, swept = false;
    for (unsigned it = 0;; ++it) {
      if (!swept) {
        swept = __all(lane >= NSWEEP || __hip_atomic_load((GAS unsigned*)(myws + OFF_CTR) + NPUSH + NGEMM + lane, AGT_RLX) == call);
        if (swept && lane < NR) __hip_atomic_store((GAS unsigned long long*)(tw + aoff + (size_t)jb.gran * 8), (unsigned long long)call << 32, SYS_RLX);
      }
      const v2u p0 = part[0], p1 = part[1], p2 = part[2], p3 = part[3], p4 = part[4], p5 = part[5];
      mx = __builtin_amdgcn_readfirstlane(umax(umax(umax(p0.x, p1.x), umax(p2.x, p3.x)), umax(p4.x, p5.x)));
      const unsigned ne = __builtin_amdgcn_readfirstlane(p0.y + p1.y);
      const unsigned nl = __builtin_amdgcn_readfirstlane(p2.y + p3.y + p4.y + p5.y);
      if (ne + nl == NR * NGEMM) break;
      if (ne >= SPEC_MIN && mx != pub) {
        if (pub == 0) STAMP(3);
        if (lane == 0) __hip_atomic_store(rp, ((unsigned long long)(2 * call) << 32) | mx, AGT_RLX);
        pub = mx;
      }
      if ((it & 63) == 63 && (__builtin_amdgcn_readfirstlane(__hip_atomic_load(tmo, AGT_RLX)) == call ||
                              __builtin_amdgcn_s_memrealtime() - t0 > DEADLINE_TICKS)) { bad = true; break; }
      __builtin_amdgcn_s_sleep(1);
    }
    if (!bad && lane == 0) {
      asm volatile("s_waitcnt vmcnt(0)" ::: "memory");
      __hip_atomic_store(rp, ((unsigned long long)(2 * call + 1) << 32) | mx, AGT_RLX);
    }
    if (lane == 0) __hip_atomic_store(ctr, call, AGT_RLX);
    STAMP(1);
    const char* gp = myws + OFF_DONE + rb * 1024 + lane * 16;
    const bool idle = lane >= 48 || (lane >> 2) == 4 + j * 2 + 1;
    while (!bad) {
      v4u a; unsigned tm;
      asm volatile("global_load_dwordx4 %0, %2, off sc0 sc1\n\t"
                   "global_load_dword %1, %3, off sc1\n\t"
                   "s_waitcnt vmcnt(0)"
                   : "=&v"(a), "=&v"(tm) : "v"(gp), "v"(tmo) : "memory");
      if (__all(idle || (a.y == call && a.w == call && a.x == mx && a.z == mx))) break;
      if (__any(tm == call) || __builtin_amdgcn_s_memrealtime() - t0 > DEADLINE_TICKS) bad = true;
      __builtin_amdgcn_s_sleep(1);
    }
    STAMP(2);
    if (bad && lane == 0) {
      __hip_atomic_store(tmo, call, AGT_RLX);
      *(GAS signed char*)(pick_ptr(p.out, me) + (size_t)me * MSH * NSH) = -128;
    }
    return;
  }
  if (!early && diag) {
    early = true;
    jb.ur0 = me * 8 + 5; jb.ur1 = me * 8 + 6; jb.ur2 = me * 8 + 7;
    jb.wsrc = me; jb.wct = rb; jb.nvalid = 12;
  }
  if (early) gemm_tile<true>(p, jb, myws, (LAS char*)smem, call, lane, wave);
  else gemm_tile<false>(p, jb, myws, (LAS char*)smem, call, lane, wave);
}

}

extern "C" void kernel_launch(int rank, int nranks, ncclComm_t comm,
                              hipStream_t stream,
                              void* const* d_in, const int* in_sizes, int n_in,
                              void* d_out, int out_size,
                              const int* in_dtypes, int out_dtype,
                              void* const* d_out_all,
                              void* const* d_ws_all, size_t ws_bytes) {
  (void)comm; (void)in_dtypes; (void)out_dtype; (void)d_out;
  if (nranks != NR || (unsigned)rank >= (unsigned)NR || n_in < 2 || in_sizes[0] != MSH * KDIM || in_sizes[1] != KDIM * NSH ||
      out_size != NR * MSH * NSH || ws_bytes < WS_NEED) {
    fprintf(stderr, "kernel_launch: unexpected problem shape\n");
    return;
  }
  Params p;
  p.x = (const char*)d_in[0];
  p.w = (const char*)d_in[1];
  p.rank = 0;
  for (int i = 0; i < NR; ++i) {
    const int r = L2P >> i * 4 & 7;
    p.ws[i] = (char*)d_ws_all[r];
    p.out[i] = (char*)d_out_all[r];
    if (r == rank) p.rank = i;
  }
  p.myws = p.ws[p.rank];
  for (int i = 0; i < NR; ++i) p.pd[i] = p.ws[PUSH_DST1(p.rank, i)];
  hipLaunchKernelGGL(ag_gemm_q8, dim3(NWG), dim3(NTHR), 0, stream, p);
  HIP_CHECK(hipGetLastError());
}
```
